```python
import numpy as np

kernel_name = "gpu_gemm_ar_m4096_k4096_n16384_e4m3e4m3_dequant"


ABSMAX_RTOL = 0.03

COMPUTE_DTYPE = "float32"
EPILOGUE = "dequant"
IN_QUANT = None
OUT_QUANT = None
IN_FP8 = "e4m3e4m3"
IN_DTYPES = {"x": "e4m3", "w_mat": "e4m3",
             "scale_x": "float32", "scale_w": "float32"}
WORLD_SIZE = 8

def _snap_fp8(x, mant_bits, max_abs, min_normal):
    a = np.abs(x.astype(np.float64))
    _m, _e = np.frexp(a)
    q = np.where(a >= min_normal, np.ldexp(1.0, _e - 1 - mant_bits),
                 min_normal * 2.0 ** -mant_bits)
    snapped = np.minimum(np.round(a / q) * q, max_abs)
    return (np.sign(x) * snapped).astype(np.float32)

def setup_inputs():

    x = _snap_fp8(np.random.randn(4096, 4096).astype(np.float32), 3, 448.0, 2.0 ** -6)
    w_mat = _snap_fp8(np.random.randn(4096, 16384).astype(np.float32), 3, 448.0, 2.0 ** -6)
    scale_x = (0.02 * (1.0 + np.random.rand(1))).astype(np.float32)
    scale_w = (0.02 * (1.0 + np.random.rand(1))).astype(np.float32)
    return {"x": x, "w_mat": w_mat, "scale_x": scale_x, "scale_w": scale_w}

def reference(x, w_mat, scale_x, scale_w):
    acc = (x @ w_mat).astype(np.float32)
    y = (acc * (scale_x * scale_w)).astype(np.float32)
    return y
```

```cpp
#include <hip/hip_runtime.h>
#include <rccl/rccl.h>
#include <cstdio>
#include <cstdint>

#define HIP_CHECK(expr) do { hipError_t _e = (expr); if (_e != hipSuccess) { \
    fprintf(stderr, "HIP error %s at %s:%d\n", hipGetErrorString(_e), __FILE__, __LINE__); \
    return; } } while (0)

namespace {

#define LAS __attribute__((address_space(3)))
typedef int i32x4 __attribute__((ext_vector_type(4)));
typedef int i32x8 __attribute__((ext_vector_type(8)));
typedef unsigned u32x4 __attribute__((ext_vector_type(4)));
typedef float f32x4 __attribute__((ext_vector_type(4)));

constexpr int M = 4096, N = 16384, NR = 8, KS = 512;
constexpr int NKS = 32;
constexpr int BLK = 32768;
constexpr int HTB = 16384;
constexpr int NMT = 16, NNT = 64;
constexpr size_t AIMG = (size_t)NMT * NKS * BLK, BIMG = (size_t)NNT * NKS * BLK, IMG = AIMG + BIMG;

constexpr int NBLK = 320;
constexpr int NTASK = NBLK * 8;
#define NCOMM 16
#define TW 4
#define OUT_AUX 2
constexpr int NWG = 256;
constexpr int NCW = (NCOMM - 1) * TW;
constexpr int GPX = (NWG - NCOMM) / 8;
constexpr int NFULL = (128 / GPX) * GPX;
constexpr int NQUART = (128 - NFULL) * 4;

constexpr size_t WS_TICKET = 0, WS_TMO = 256, WS_PROG = 512, WS_CLAIM = 768, WS_QCLAIM = 1024, WS_DONE = 4096, WS_IMG = 1u << 20;
constexpr size_t WS_STAMP = 65536;
constexpr size_t WS_NEED = WS_IMG + 2 * IMG;
#define STAMP(p, k) do { } while (0)
constexpr unsigned long long DEADLINE_TICKS = 50000000ull;

constexpr int LDS_CTRL = 8 * HTB;
constexpr int LDS_BYTES = LDS_CTRL + 64;

struct Params {
    const unsigned char* x;
    const unsigned char* w;
    const float* sx;
    const float* sw;
    float* out;
    unsigned char* ws[NR];
    int rank;
};

#define GATE_ON 1

__device__ __forceinline__ int need_base(int j) { return (j < 16 ? 16 : 128 + ((j >> 4) - 1) * 64) + (j & 1) + 1; }
__device__ __forceinline__ int need_step(int j) { return j < 16 ? 32 : 16; }

__device__ __forceinline__ void transpose4(unsigned d0, unsigned d1, unsigned d2, unsigned d3,
                                           unsigned& o0, unsigned& o1, unsigned& o2, unsigned& o3) {
    const unsigned t0 = __builtin_amdgcn_perm(d1, d0, 0x05010400u), t1 = __builtin_amdgcn_perm(d1, d0, 0x07030602u);
    const unsigned t2 = __builtin_amdgcn_perm(d3, d2, 0x05010400u), t3 = __builtin_amdgcn_perm(d3, d2, 0x07030602u);
    o0 = __builtin_amdgcn_perm(t2, t0, 0x05040100u); o1 = __builtin_amdgcn_perm(t2, t0, 0x07060302u);
    o2 = __builtin_amdgcn_perm(t3, t1, 0x05040100u); o3 = __builtin_amdgcn_perm(t3, t1, 0x07060302u);
}

__device__ __forceinline__ void comm_tasks(const Params& p, unsigned call, int cw, int lane) {
    const size_t img = WS_IMG + (size_t)(call & 1u) * IMG;
    __amdgpu_buffer_rsrc_t rs[NR];
#pragma unroll
    for (int d = 0; d < NR; ++d) rs[d] = __builtin_amdgcn_make_buffer_rsrc(p.ws[d] + img, 0, (int)IMG, 0x00020000);
    unsigned* done = (unsigned*)(p.ws[NR - 1] + WS_DONE);
    const int r = lane & 15, g = lane >> 4;
    unsigned* claim = (unsigned*)(p.ws[NR - 1] + WS_CLAIM);
    const unsigned cbase = (call - 1u) * (unsigned)NTASK;
#define CLAIM_AND_DRAIN() do { if (lane == 0) asm volatile("global_atomic_add %0, %1, %0, off sc0\n\ts_waitcnt vmcnt(0)" : "+v"(tk) : "v"(claim) : "memory"); } while (0)
    int tprev = -1;
    for (int t = cw; (unsigned)t < (unsigned)NTASK; ) {
        unsigned tk = 1u;
        const int bp = t >> 3, sub = t & 7;
        int kk, tile; bool is_a;
        if (bp < 128) { kk = bp >> 5; is_a = !(bp & 16); tile = is_a ? (bp & 15) : ((bp - 2 * p.rank) & 15); }
        else { const int q = bp - 128; kk = (q >> 4) & 3; is_a = false; tile = 16 + (q >> 6) * 16 + ((q - 2 * p.rank) & 15); }
        const int s = p.rank * 4 + kk;
        u32x4 o[4]; unsigned off[4];
        if (is_a) {
            const unsigned char* src = p.x + (size_t)(256 * tile + 32 * sub + r) * KS + 128 * kk + 16 * g;
#pragma unroll
            for (int q = 0; q < 4; ++q) o[q] = *(const u32x4*)(src + (q >> 1) * 16 * KS + (q & 1) * 64);
            const unsigned base = (unsigned)(tile * NKS + s) * BLK + sub * 4096 + lane * 16;
#pragma unroll
            for (int q = 0; q < 4; ++q) off[q] = base + q * 1024;
            CLAIM_AND_DRAIN();
            asm volatile("" : "+v"(o[0]), "+v"(o[1]), "+v"(o[2]), "+v"(o[3]) :: "memory");
        } else {
            const int G = sub >> 1, h = sub & 1;
            const unsigned char* src = p.w + (size_t)(128 * kk + 64 * h + 16 * g) * N + 256 * tile + 64 * G + 4 * r;
            unsigned d[16];
#pragma unroll
            for (int b = 0; b < 16; ++b) d[b] = *(const unsigned*)(src + (size_t)b * N);
            CLAIM_AND_DRAIN();
#pragma unroll
            for (int a = 0; a < 4; ++a) {
                unsigned o0, o1, o2, o3;
                transpose4(d[4 * a], d[4 * a + 1], d[4 * a + 2], d[4 * a + 3], o0, o1, o2, o3);
                o[0][a] = o0; o[1][a] = o1; o[2][a] = o2; o[3][a] = o3;
            }
            const unsigned base = (unsigned)AIMG + (unsigned)(tile * NKS + s) * BLK + G * 4096 + h * 1024 + lane * 16;
#pragma unroll
            for (int q = 0; q < 4; ++q) off[q] = base + (q >> 1) * HTB + (q & 1) * 2048;
        }
        const int tnext = NCW + (int)((unsigned)__builtin_amdgcn_readfirstlane((int)tk) - cbase);
        if (tprev >= 0 && lane == 0) __hip_atomic_store(done + tprev, call, __ATOMIC_RELAXED, __HIP_MEMORY_SCOPE_AGENT);
#pragma unroll
        for (int q = 0; q < 4; ++q)
#pragma unroll
            for (int d = 0; d < NR; ++d) __builtin_amdgcn_raw_buffer_store_b128(o[q], rs[d], off[q], 0, 17);
        tprev = t;
        t = tnext;
    }
    asm volatile("s_waitcnt vmcnt(0)" ::: "memory");
    if (tprev >= 0 && lane == 0) __hip_atomic_store(done + tprev, call, __ATOMIC_RELAXED, __HIP_MEMORY_SCOPE_AGENT);
    STAMP(p, 8 + (cw % TW));
#undef CLAIM_AND_DRAIN
}

__device__ __forceinline__ void comm_publish(const Params& p, unsigned call, int lane) {
    const unsigned* done = (const unsigned*)(p.ws[NR - 1] + WS_DONE);
    unsigned* tmo = (unsigned*)(p.ws[NR - 1] + WS_TMO);
    unsigned* dst = (unsigned*)(p.ws[NR - 1] + WS_PROG);
#pragma unroll
    for (int d = 0; d < NR - 1; ++d) if (lane == d) dst = (unsigned*)(p.ws[d] + WS_PROG);
    dst += p.rank;
    const unsigned base = (call - 1u) * NBLK;
    const unsigned long long t0 = __builtin_amdgcn_s_memrealtime();
    int nt = 0, pub = 0;
    while (nt < NTASK) {
        const int idx = nt + lane;
        unsigned v = 0;
        if (idx < NTASK) v = __hip_atomic_load(done + idx, __ATOMIC_RELAXED, __HIP_MEMORY_SCOPE_AGENT);
        const unsigned long long miss = ~__ballot(v == call);
        const int n = miss ? __builtin_ctzll(miss) : 64;
        nt += n;
        if ((nt >> 3) > pub) {
            pub = nt >> 3;
            if (lane < NR) __hip_atomic_store(dst, base + (unsigned)pub, __ATOMIC_RELAXED, __HIP_MEMORY_SCOPE_SYSTEM);
        }
        if (n == 0) {
            if (__builtin_amdgcn_s_memrealtime() - t0 > DEADLINE_TICKS) {
                if (lane == 0) __hip_atomic_store(tmo, call, __ATOMIC_RELAXED, __HIP_MEMORY_SCOPE_AGENT);
                break;
            }
            __builtin_amdgcn_s_sleep(2);
        }
    }
}

struct Tile { int i, j, need0, step, rot; };

__device__ __forceinline__ void tile_of(int xcd, int e, Tile& t) {
    t.i = 4 * (xcd & 3) + (e & 3);
    t.j = 2 * (e >> 2) + (xcd >> 2);
    t.need0 = need_base(t.j);
    t.step = need_step(t.j);
    t.rot = 8 - ((t.j & 15) >> 1);
}

__device__ __forceinline__ bool next_tile(int xcd, int r, int n, Tile& t) {
    const int e = r + n * GPX;
    if (e >= NFULL) return false;
    tile_of(xcd, e, t);
    return true;
}

__device__ __forceinline__ void gemm_role(const Params& p, unsigned call, LAS unsigned char* lds, int xcd, int rk) {
    const int tid = threadIdx.x, wid = __builtin_amdgcn_readfirstlane(tid >> 6), lane = tid & 63;
    const int wr = wid >> 2, wc = wid & 3;
    const unsigned char* aimg = p.ws[NR - 1] + WS_IMG + (size_t)(call & 1u) * IMG;
    const unsigned char* bimg = aimg + AIMG;
    const unsigned* prog = (const unsigned*)(p.ws[NR - 1] + WS_PROG);
    unsigned* tmo = (unsigned*)(p.ws[NR - 1] + WS_TMO);
    LAS volatile unsigned* ctrl = (LAS volatile unsigned*)(lds + LDS_CTRL);
    const unsigned pbase = (call - 1u) * NBLK;
    unsigned voff0 = (unsigned)tid * 16u, voff1 = voff0 + 8192u;
    asm volatile("" : "+v"(voff0), "+v"(voff1));
    const unsigned ldsw = (unsigned)wid * 1024u;
    int aoff = wr * 8192 + lane * 16, boff = 4 * HTB + wc * 4096 + lane * 16;
    asm volatile("" : "+v"(aoff), "+v"(boff));
    unsigned vprog = pbase;
    const unsigned* progl = prog + (lane & 7);

#define SA(b, h) (((b) * 2 + (h)) * HTB)
#define SB(b, h) ((4 + (b) * 2 + (h)) * HTB)
#define STAGE(bufoff, gbase) do { const unsigned char* _g = (gbase); \
        __builtin_amdgcn_global_load_lds((const unsigned*)(_g + voff0), (LAS unsigned*)(lds + (bufoff) + ldsw), 16, 0, 0); \
        __builtin_amdgcn_global_load_lds((const unsigned*)(_g + voff1), (LAS unsigned*)(lds + (bufoff) + ldsw + 8192), 16, 0, 0); } while (0)
#define FRAG(off) __builtin_shufflevector(*(const LAS i32x4*)(lds + (off)), *(const LAS i32x4*)(lds + (off) + 1024), 0, 1, 2, 3, 4, 5, 6, 7)
#define LDA(dst, b, h) do { _Pragma("unroll") for (int m = 0; m < 4; ++m) dst[m] = FRAG(SA(b, h) + aoff + m * 2048); } while (0)
#define LDB(dst, b, h) do { _Pragma("unroll") for (int n = 0; n < 2; ++n) dst[n] = FRAG(SA(b, h) + boff + n * 2048); } while (0)
#define MMA(ai, bj, At, Bt) do { __builtin_amdgcn_s_setprio(1); _Pragma("unroll") for (int m = 0; m < 4; ++m) _Pragma("unroll") for (int n = 0; n < 2; ++n) \
        acc[ai][bj][m][n] = __builtin_amdgcn_mfma_scale_f32_16x16x128_f8f6f4(At[m], Bt[n], acc[ai][bj][m][n], 0, 0, 0, 0, 0, 0); __builtin_amdgcn_s_setprio(0); } while (0)
#define WAIT_V(n) asm volatile("s_waitcnt vmcnt(" #n ")" ::: "memory")
#define WAIT_L(n) asm volatile("s_waitcnt lgkmcnt(" #n ")" ::: "memory")
#define BAR __builtin_amdgcn_s_barrier()
#define SCHED __builtin_amdgcn_sched_barrier(0)
#define KOFF(rot, t) ((size_t)(((((rot) + (t)) & 7) * 4 + ((t) >> 3)) * BLK))
#define GATE(need0, step, rot, t) do { if (GATE_ON && wid == 0) { \
        const int _src = ((rot) + (t)) & 7; \
        const unsigned _need = pbase + (unsigned)((need0) + ((t) >> 3) * (step) + ((t) & 7) * 2); \
        if ((int)((unsigned)__builtin_amdgcn_readlane((int)vprog, _src) - _need) < 0) { \
            const unsigned long long _t0 = __builtin_amdgcn_s_memrealtime(); \
            for (;;) { \
                vprog = __hip_atomic_load(progl, __ATOMIC_RELAXED, __HIP_MEMORY_SCOPE_SYSTEM); \
                if ((int)((unsigned)__builtin_amdgcn_readlane((int)vprog, _src) - _need) >= 0) break; \
                if (__hip_atomic_load(tmo, __ATOMIC_RELAXED, __HIP_MEMORY_SCOPE_AGENT) == call || \
                    __builtin_amdgcn_s_memrealtime() - _t0 > DEADLINE_TICKS) { \
                    __hip_atomic_store(tmo, call, __ATOMIC_RELAXED, __HIP_MEMORY_SCOPE_AGENT); \
                    ctrl[1] = 1u; break; } \
                __builtin_amdgcn_s_sleep(4); \
            } \
            __builtin_amdgcn_s_waitcnt(0xC07F); \
            asm volatile("" ::: "memory"); \
        } } } while (0)
#define POLL_ISSUE() do { if (wid == 0) asm volatile("global_load_dword %0, %1, off sc0 sc1" : "+v"(vprog) : "v"(progl) : "memory"); } while (0)
#define POLL_LANDED() asm volatile("" : "+v"(vprog))

    Tile cur, nxt;
    int un = 0;
    if (!next_tile(xcd, rk, 0, cur)) return;
    f32x4 acc[2][2][4][2];
#pragma unroll
    for (int a = 0; a < 2; ++a)
#pragma unroll
        for (int b = 0; b < 2; ++b)
#pragma unroll
            for (int m = 0; m < 4; ++m)
#pragma unroll
                for (int n = 0; n < 2; ++n) acc[a][b][m][n] = (f32x4){0.f, 0.f, 0.f, 0.f};
    i32x8 At[4], B0[2], B1[2];
    const unsigned char* cA = aimg + (size_t)cur.i * NKS * BLK;
    const unsigned char* cB = bimg + (size_t)cur.j * NKS * BLK;
    const float scale = p.sx[0] * p.sw[0];

    GATE(cur.need0, cur.step, cur.rot, 0); GATE(cur.need0, cur.step, cur.rot, 1);
    BAR;
    STAGE(SB(0, 0), cB + KOFF(cur.rot, 0)); STAGE(SA(0, 0), cA + KOFF(cur.rot, 0)); STAGE(SB(0, 1), cB + KOFF(cur.rot, 0) + HTB); STAGE(SA(0, 1), cA + KOFF(cur.rot, 0) + HTB);
    if (wr == 1) BAR;
    WAIT_V(4); BAR;
    STAGE(SB(1, 0), cB + KOFF(cur.rot, 1)); STAGE(SA(1, 0), cA + KOFF(cur.rot, 1)); STAGE(SB(1, 1), cB + KOFF(cur.rot, 1) + HTB);
    WAIT_V(6); BAR;
    if (wid == 0) STAMP(p, 1);
    for (;;) {
        const bool has_next = next_tile(xcd, rk, un + 1, nxt);
        if (!has_next) nxt = cur;
        const unsigned char* nA = aimg + (size_t)nxt.i * NKS * BLK;
        const unsigned char* nB = bimg + (size_t)nxt.j * NKS * BLK;
        for (int t = 0; t < NKS; t += 2) {
            const bool last = (t == NKS - 2);
            const int t2 = (t + 2) & (NKS - 1), t3 = t2 + 1;
            const int gn = last ? nxt.need0 : cur.need0, gs = last ? nxt.step : cur.step, gr = last ? nxt.rot : cur.rot;
            const unsigned char* a1 = cA + KOFF(cur.rot, t + 1);
            const unsigned char* a2 = (last ? nA : cA) + KOFF(gr, t2);
            const unsigned char* b2 = (last ? nB : cB) + KOFF(gr, t2);
            const unsigned char* a3 = (last ? nA : cA) + KOFF(gr, t3);
            const unsigned char* b3 = (last ? nB : cB) + KOFF(gr, t3);
            GATE(gn, gs, gr, t2);
            POLL_ISSUE();
            LDB(B0, 0, 0); SCHED; LDA(At, 0, 0); STAGE(SA(1, 1), a1 + HTB);
            WAIT_L(8); BAR; WAIT_L(0); MMA(0, 0, At, B0); BAR; SCHED;
            LDB(B1, 0, 1); STAGE(SB(0, 0), b2);
            BAR; WAIT_L(0); MMA(0, 1, At, B1); BAR;
            LDA(At, 0, 1); STAGE(SA(0, 0), a2);
            BAR; WAIT_L(0); MMA(1, 0, At, B0); BAR; SCHED;
            STAGE(SB(0, 1), b2 + HTB);
            WAIT_V(6); POLL_LANDED(); BAR; MMA(1, 1, At, B1); BAR;
            GATE(gn, gs, gr, t3);
            POLL_ISSUE();
            LDB(B0, 1, 0); SCHED; LDA(At, 1, 0); STAGE(SA(0, 1), a2 + HTB);
            WAIT_L(8); BAR; WAIT_L(0); MMA(0, 0, At, B0); BAR; SCHED;
            LDB(B1, 1, 1); STAGE(SB(1, 0), b3);
            BAR; WAIT_L(0); MMA(0, 1, At, B1); BAR;
            LDA(At, 1, 1); STAGE(SA(1, 0), a3);
            BAR; WAIT_L(0); MMA(1, 0, At, B0); BAR; SCHED;
            STAGE(SB(1, 1), b3 + HTB);
            WAIT_V(6); POLL_LANDED(); BAR; MMA(1, 1, At, B1); BAR;
        }
        if (wid == 0) STAMP(p, 2 + 2 * un);
        {
            int tz = threadIdx.x; asm volatile("" : "+v"(tz));
            const int l = tz & 63, c = l & 15, g = l >> 4;
            const float sc = ctrl[1] ? __builtin_nanf("") : scale;
            const __amdgpu_buffer_rsrc_t ro = __builtin_amdgcn_make_buffer_rsrc(p.out + (size_t)(cur.i * 256 + wr * 64) * N + cur.j * 256 + wc * 64, 0, 192 * N * 4, 0x00020000);
            const int vo = g * (4 * N * 4) + c * 16;
#pragma unroll
            for (int ai = 0; ai < 2; ++ai)
#pragma unroll
                for (int m = 0; m < 4; ++m) {
#pragma unroll
                    for (int e = 0; e < 4; ++e) {
                        const f32x4 v = {acc[ai][0][m][0][e] * sc, acc[ai][0][m][1][e] * sc, acc[ai][1][m][0][e] * sc, acc[ai][1][m][1][e] * sc};
                        __builtin_amdgcn_raw_buffer_store_b128(__builtin_bit_cast(u32x4, v), ro, vo + (ai * 128 + m * 16 + e) * (N * 4), 0, OUT_AUX);
                    }
                }
        }
        if (wid == 0) STAMP(p, 3 + 2 * un);
        if (!has_next) break;
#pragma unroll
        for (int a = 0; a < 2; ++a)
#pragma unroll
            for (int b = 0; b < 2; ++b)
#pragma unroll
                for (int m = 0; m < 4; ++m)
#pragma unroll
                    for (int n = 0; n < 2; ++n) acc[a][b][m][n] = (f32x4){0.f, 0.f, 0.f, 0.f};
        cur = nxt; cA = nA; cB = nB; ++un;
    }
    WAIT_V(0);
    if (wr == 0) BAR;
#undef SA
#undef SB
#undef STAGE
#undef FRAG
#undef LDA
#undef LDB
#undef MMA
#undef WAIT_V
#undef WAIT_L
#undef BAR
#undef SCHED
#undef KOFF
#undef GATE
#undef POLL_ISSUE
#undef POLL_LANDED
}

__device__ __forceinline__ void quarter_role(const Params& p, unsigned call, LAS unsigned char* lds, int xcd, int qd) {
    const int tid = threadIdx.x, wid = __builtin_amdgcn_readfirstlane(tid >> 6), lane = tid & 63;
    const int wm = wid >> 1, wg = wid & 1;
    const int qa = (qd >> 1) & 1, qb = qd & 1;
    Tile tl; tile_of(xcd, NFULL + (qd >> 2), tl);
    const int ti = tl.i, tj = tl.j, rot = tl.rot;
    const unsigned char* aimg = p.ws[NR - 1] + WS_IMG + (size_t)(call & 1u) * IMG;
    const unsigned char* cA = aimg + (size_t)ti * NKS * BLK + qa * HTB;
    const unsigned char* cB = aimg + AIMG + (size_t)tj * NKS * BLK + qb * 8192;
    LAS volatile unsigned* ctrl = (LAS volatile unsigned*)(lds + LDS_CTRL);
    if (GATE_ON && wid == 0) {
        const unsigned* progl = (const unsigned*)(p.ws[NR - 1] + WS_PROG) + (lane & 7);
        unsigned* tmo = (unsigned*)(p.ws[NR - 1] + WS_TMO);
        const unsigned need = (call - 1u) * NBLK + (unsigned)(tl.need0 + 3 * tl.step + (((lane & 7) - rot) & 7) * 2);
        const unsigned long long t0 = __builtin_amdgcn_s_memrealtime();
        for (;;) {
            const unsigned v = __hip_atomic_load(progl, __ATOMIC_RELAXED, __HIP_MEMORY_SCOPE_SYSTEM);
            if (__all((int)(v - need) >= 0)) break;
            if (__hip_atomic_load(tmo, __ATOMIC_RELAXED, __HIP_MEMORY_SCOPE_AGENT) == call ||
                __builtin_amdgcn_s_memrealtime() - t0 > DEADLINE_TICKS) {
                __hip_atomic_store(tmo, call, __ATOMIC_RELAXED, __HIP_MEMORY_SCOPE_AGENT);
                ctrl[1] = 1u; break; }
            __builtin_amdgcn_s_sleep(4);
        }
        __builtin_amdgcn_s_waitcnt(0xC07F);
    }
    asm volatile("s_waitcnt vmcnt(0) lgkmcnt(0)" ::: "memory");
    __builtin_amdgcn_s_barrier();
    asm volatile("" ::: "memory");
    unsigned voff = (unsigned)tid * 16u;
    asm volatile("" : "+v"(voff));
    const unsigned ldsw = (unsigned)wid * 1024u;
    int aoff = wm * 4096 + lane * 16, boff = HTB + wg * 4096 + lane * 16;
    asm volatile("" : "+v"(aoff), "+v"(boff));
#define QKOFF(t) ((size_t)((((rot + (t)) & 7) * 4 + ((t) >> 3)) * BLK))
#define QSTAGE(t) do { const int _t = (t) & (NKS - 1); const unsigned char* _a = cA + QKOFF(_t); const unsigned char* _b = cB + QKOFF(_t); \
        LAS unsigned char* _l = lds + ((t) & 3) * BLK + ldsw; \
        __builtin_amdgcn_global_load_lds((const unsigned*)(_a + voff), (LAS unsigned*)(_l), 16, 0, 0); \
        __builtin_amdgcn_global_load_lds((const unsigned*)(_a + 8192 + voff), (LAS unsigned*)(_l + 8192), 16, 0, 0); \
        __builtin_amdgcn_global_load_lds((const unsigned*)(_b + voff), (LAS unsigned*)(_l + HTB), 16, 0, 0); \
        __builtin_amdgcn_global_load_lds((const unsigned*)(_b + HTB + voff), (LAS unsigned*)(_l + HTB + 8192), 16, 0, 0); } while (0)
#define QFRAG(off) __builtin_shufflevector(*(const LAS i32x4*)(lds + (off)), *(const LAS i32x4*)(lds + (off) + 1024), 0, 1, 2, 3, 4, 5, 6, 7)
#define QREAD(A, B, slot) do { _Pragma("unroll") for (int m = 0; m < 2; ++m) A[m] = QFRAG((slot) * BLK + aoff + m * 2048); \
        _Pragma("unroll") for (int q = 0; q < 4; ++q) B[q] = QFRAG((slot) * BLK + boff + (q >> 1) * 8192 + (q & 1) * 2048); \
        __builtin_amdgcn_sched_barrier(0); } while (0)
#define QSYNC(A, B) do { __builtin_amdgcn_sched_barrier(0); asm volatile("s_waitcnt vmcnt(8) lgkmcnt(0)" : "+v"(A[0]), "+v"(A[1]), "+v"(B[0]), "+v"(B[1]), "+v"(B[2]), "+v"(B[3]) :: "memory"); \
        __builtin_amdgcn_sched_barrier(0); __builtin_amdgcn_s_barrier(); asm volatile("" ::: "memory"); } while (0)
    f32x4 acc[2][4];
#pragma unroll
    for (int m = 0; m < 2; ++m)
#pragma unroll
        for (int q = 0; q < 4; ++q) acc[m][q] = (f32x4){0.f, 0.f, 0.f, 0.f};
    i32x8 A0[2], B0[4], A1[2], B1[4];
    QSTAGE(0); QSTAGE(1); QSTAGE(2);
    asm volatile("s_waitcnt vmcnt(8)" ::: "memory");
    __builtin_amdgcn_s_barrier();
    asm volatile("" ::: "memory");
    QSTAGE(3);
    QREAD(A0, B0, 0);
#define QSB __builtin_amdgcn_sched_barrier(0)
#define QM(A, B, m, q) acc[m][q] = __builtin_amdgcn_mfma_scale_f32_16x16x128_f8f6f4(A[m], B[q], acc[m][q], 0, 0, 0, 0, 0, 0); QSB
#define QSTEP(A, B, An, Bn, slot, t) do { \
        QSYNC(A, B); \
        const int _t = (t) & (NKS - 1); const unsigned char* _a = cA + QKOFF(_t); const unsigned char* _b = cB + QKOFF(_t); \
        LAS unsigned char* _l = lds + ((t) & 3) * BLK + ldsw; \
        QM(A, B, 0, 0); An[0] = QFRAG((slot) * BLK + aoff); Bn[0] = QFRAG((slot) * BLK + boff); QSB; \
        QM(A, B, 0, 1); An[1] = QFRAG((slot) * BLK + aoff + 2048); Bn[1] = QFRAG((slot) * BLK + boff + 2048); QSB; \
        QM(A, B, 0, 2); Bn[2] = QFRAG((slot) * BLK + boff + 8192); Bn[3] = QFRAG((slot) * BLK + boff + 8192 + 2048); QSB; \
        QM(A, B, 0, 3); __builtin_amdgcn_global_load_lds((const unsigned*)(_a + voff), (LAS unsigned*)(_l), 16, 0, 0); QSB; \
        QM(A, B, 1, 0); __builtin_amdgcn_global_load_lds((const unsigned*)(_a + 8192 + voff), (LAS unsigned*)(_l + 8192), 16, 0, 0); QSB; \
        QM(A, B, 1, 1); __builtin_amdgcn_global_load_lds((const unsigned*)(_b + voff), (LAS unsigned*)(_l + HTB), 16, 0, 0); QSB; \
        QM(A, B, 1, 2); __builtin_amdgcn_global_load_lds((const unsigned*)(_b + HTB + voff), (LAS unsigned*)(_l + HTB + 8192), 16, 0, 0); QSB; \
        QM(A, B, 1, 3); } while (0)
#pragma unroll 1
    for (int t = 0; t < NKS; t += 4) {
        QSTEP(A0, B0, A1, B1, 1, t + 4);
        QSTEP(A1, B1, A0, B0, 2, t + 5);
        QSTEP(A0, B0, A1, B1, 3, t + 6);
        QSTEP(A1, B1, A0, B0, 0, t + 7);
    }
#undef QSB
#undef QM
#undef QSTEP
    const int c = lane & 15, g = lane >> 4;
    const float sc = ctrl[1] ? __builtin_nanf("") : p.sx[0] * p.sw[0];
    const __amdgpu_buffer_rsrc_t ro = __builtin_amdgcn_make_buffer_rsrc(p.out + (size_t)(ti * 256 + qa * 128 + wm * 32) * N + tj * 256 + (2 * qb + wg) * 64, 0, 32 * N * 4, 0x00020000);
    const int vo = g * (4 * N * 4) + c * 16;
#pragma unroll
    for (int m = 0; m < 2; ++m)
#pragma unroll
        for (int r = 0; r < 4; ++r) {
            const f32x4 v = {acc[m][0][r] * sc, acc[m][1][r] * sc, acc[m][2][r] * sc, acc[m][3][r] * sc};
            __builtin_amdgcn_raw_buffer_store_b128(__builtin_bit_cast(u32x4, v), ro, vo + (m * 16 + r) * (N * 4), 0, OUT_AUX);
        }
    asm volatile("s_waitcnt vmcnt(0) lgkmcnt(0)" ::: "memory");
#undef QKOFF
#undef QSTAGE
#undef QFRAG
#undef QREAD
#undef QSYNC
}

__global__ void __launch_bounds__(512, 2) fused_gemm_allgather(const Params p) {
    extern __shared__ __attribute__((aligned(16))) unsigned char shm[];
    LAS unsigned char* lds = (LAS unsigned char*)shm;
    LAS volatile unsigned* ctrl = (LAS volatile unsigned*)(lds + LDS_CTRL);
    if (threadIdx.x == 0) {
        const unsigned ticket = __hip_atomic_fetch_add((unsigned*)(p.ws[NR - 1] + WS_TICKET), 1u, __ATOMIC_RELAXED, __HIP_MEMORY_SCOPE_AGENT);
        ctrl[0] = ticket / NWG + 1u;
        ctrl[1] = 0u;
        __builtin_amdgcn_fence(__ATOMIC_ACQUIRE, "");
        asm volatile("s_waitcnt vmcnt(0)" ::: "memory");
    }
    __syncthreads();
    const unsigned call = (unsigned)__builtin_amdgcn_readfirstlane((int)ctrl[0]);
    const int b = blockIdx.x;
    if (threadIdx.x == 0) STAMP(p, 0);
    const int xcd = b & 7;
    if (b < NCOMM) {
        const int wid = __builtin_amdgcn_readfirstlane((int)(threadIdx.x >> 6)), lane = threadIdx.x & 63;
        bool run = true;
        if (run) {
            if (b == 0) { if (wid == 0) comm_publish(p, call, lane); }
            else if (wid < TW) comm_tasks(p, call, (b - 1) * TW + wid, lane);
        }
    } else {
        gemm_role(p, call, lds, xcd, (b - NCOMM) >> 3);
    }
    unsigned* qclaim = (unsigned*)(p.ws[NR - 1] + WS_QCLAIM) + xcd * 32;
    const unsigned qbase = (call - 1u) * (unsigned)(NQUART + NWG / 8);
    for (;;) {
        asm volatile("s_waitcnt vmcnt(0) lgkmcnt(0)" ::: "memory");
        __builtin_amdgcn_s_barrier();
        if (threadIdx.x == 0) ctrl[2] = __hip_atomic_fetch_add(qclaim, 1u, __ATOMIC_RELAXED, __HIP_MEMORY_SCOPE_AGENT) - qbase;
        asm volatile("s_waitcnt vmcnt(0) lgkmcnt(0)" ::: "memory");
        __builtin_amdgcn_s_barrier();
        asm volatile("" ::: "memory");
        const unsigned qd = (unsigned)__builtin_amdgcn_readfirstlane((int)ctrl[2]);
        if (qd >= (unsigned)NQUART) break;
        quarter_role(p, call, lds, xcd, (int)qd);
    }
    if (threadIdx.x == 0) STAMP(p, 12);
}

}

extern "C" void kernel_launch(int rank, int nranks, ncclComm_t comm,
                              hipStream_t stream,
                              void* const* d_in, const int* in_sizes, int n_in,
                              void* d_out, int out_size,
                              const int* in_dtypes, int out_dtype,
                              void* const* d_out_all,
                              void* const* d_ws_all, size_t ws_bytes) {
    (void)comm; (void)in_sizes; (void)in_dtypes; (void)out_dtype; (void)d_out_all;
    if (nranks != NR || n_in != 4 || (unsigned)rank >= (unsigned)NR || ws_bytes < WS_NEED || out_size != M * N) {
        fprintf(stderr, "kernel_launch: unexpected problem shape\n");
        return;
    }
    Params p;
    p.x = (const unsigned char*)d_in[0];
    p.w = (const unsigned char*)d_in[1];
    p.sx = (const float*)d_in[2];
    p.sw = (const float*)d_in[3];
    p.out = (float*)d_out;
    for (int i = 0; i < NR; ++i) p.ws[i] = (unsigned char*)d_ws_all[(rank + 1 + i) % NR];
    p.rank = rank;
    HIP_CHECK(hipFuncSetAttribute((const void*)fused_gemm_allgather, hipFuncAttributeMaxDynamicSharedMemorySize, LDS_BYTES));
    hipLaunchKernelGGL(fused_gemm_allgather, dim3(NWG), dim3(512), LDS_BYTES, stream, p);
    HIP_CHECK(hipGetLastError());
}
```

```cpp
#include <hip/hip_runtime.h>
#include <rccl/rccl.h>
#include <cstdio>
#include <cstdint>

#define HIP_CHECK(expr) do { hipError_t _e = (expr); if (_e != hipSuccess) { \
    fprintf(stderr, "HIP error %s at %s:%d\n", hipGetErrorString(_e), __FILE__, __LINE__); \
    return; } } while (0)

namespace {

#define LAS __attribute__((address_space(3)))
typedef int i32x4 __attribute__((ext_vector_type(4)));
typedef int i32x8 __attribute__((ext_vector_type(8)));
typedef unsigned u32x4 __attribute__((ext_vector_type(4)));
typedef float f32x4 __attribute__((ext_vector_type(4)));

constexpr int M = 4096, N = 16384, NR = 8, KS = 512;
constexpr int NKS = 32;
constexpr int BLK = 32768;
constexpr int HTB = 16384;
constexpr int NMT = 16, NNT = 64, NREG = 60;
constexpr size_t AIMG = (size_t)NMT * NKS * BLK, BIMG = (size_t)NNT * NKS * BLK, IMG = AIMG + BIMG;

constexpr int NBLK_IN = 320, NBLK = 384;
constexpr int NTASK_IN = NBLK_IN * 8, NTASK = NBLK * 8;
#define TW 3
#define TW0 1
#define OUT_AUX 2
constexpr int NWG = 256, NCOMM = 14;
constexpr int NCW = (NCOMM - 1) * TW + TW0;

constexpr size_t WS_TICKET = 0, WS_EPOCH = 128, WS_TMO = 256, WS_PROG = 512, WS_CLAIM = 768, WS_LCNT = 896, WS_TCNT = 1024, WS_DONE = 4096, WS_IMG = 1u << 20;
constexpr size_t WS_STAMP = 65536;
constexpr size_t WS_NEED = WS_IMG + 2 * IMG;
#define STAMP(p, k) do { } while (0)
constexpr unsigned long long DEADLINE_TICKS = 50000000ull;

constexpr int LDS_CTRL = 8 * HTB;
constexpr int LDS_BYTES = LDS_CTRL + 64;

struct Params {
    const unsigned char* x;
    const unsigned char* w;
    const float* sx;
    const float* sw;
    unsigned char* ws[NR];
    float* out[NR];
    int rank;
};

#define GATE_ON 1

__device__ __forceinline__ void transpose4(unsigned d0, unsigned d1, unsigned d2, unsigned d3,
                                           unsigned& o0, unsigned& o1, unsigned& o2, unsigned& o3) {
    const unsigned t0 = __builtin_amdgcn_perm(d1, d0, 0x05010400u), t1 = __builtin_amdgcn_perm(d1, d0, 0x07030602u);
    const unsigned t2 = __builtin_amdgcn_perm(d3, d2, 0x05010400u), t3 = __builtin_amdgcn_perm(d3, d2, 0x07030602u);
    o0 = __builtin_amdgcn_perm(t2, t0, 0x05040100u); o1 = __builtin_amdgcn_perm(t2, t0, 0x07060302u);
    o2 = __builtin_amdgcn_perm(t3, t1, 0x05040100u); o3 = __builtin_amdgcn_perm(t3, t1, 0x07060302u);
}

__device__ __forceinline__ unsigned load_epoch(const Params& p) {
    return __hip_atomic_load((const unsigned*)(p.ws[NR - 1] + WS_EPOCH), __ATOMIC_RELAXED, __HIP_MEMORY_SCOPE_AGENT);
}

struct Task { int kind, only, tile, kk, sub; unsigned base; };

__device__ __forceinline__ void task_decode(int rank, int t, int lane, Task& k) {
    const int bp = t >> 3;
    k.sub = t & 7; k.only = -1; k.kind = 1;
    if (bp < 144) {
        k.kk = bp / 36; const int q = bp - k.kk * 36;
        if (q < 4) { k.tile = NREG + q; k.only = q; } else { k.kind = q & 1; k.tile = (q - 4) >> 1; }
    } else if (bp < 272) { const int q = bp - 144; k.kk = (q >> 4) & 3; k.tile = 16 + (q >> 6) * 16 + (q & 15); }
    else if (bp < NBLK_IN) { const int q = bp - 272; k.kk = q / 12; k.tile = 48 + (q - k.kk * 12); }
    else { k.kind = 2; k.kk = 0; k.tile = (bp - NBLK_IN) >> 3; }
    const unsigned blk = (unsigned)(k.tile * NKS + rank * 4 + k.kk) * BLK + lane * 16;
    k.base = k.kind == 0 ? blk + k.sub * 4096
           : k.kind == 1 ? (unsigned)AIMG + blk + (k.sub >> 1) * 4096 + (k.sub & 1) * 1024
                         : (unsigned)((t - NTASK_IN) * 4) * (unsigned)(N * 4) + lane * 16;
}

__device__ __forceinline__ void task_load(const Params& p, const Task& k, int lane, unsigned (&raw)[16]) {
    const int r = lane & 15, g = lane >> 4;
    if (k.kind == 0) {
        const unsigned char* src = p.x + (size_t)(256 * k.tile + 32 * k.sub + r) * KS + 128 * k.kk + 16 * g;
#pragma unroll
        for (int q = 0; q < 4; ++q) {
            const u32x4 v = *(const u32x4*)(src + (q >> 1) * 16 * KS + (q & 1) * 64);
            raw[4 * q] = v[0]; raw[4 * q + 1] = v[1]; raw[4 * q + 2] = v[2]; raw[4 * q + 3] = v[3];
        }
    } else {
        const unsigned char* src = p.w + (size_t)(128 * k.kk + 64 * (k.sub & 1) + 16 * g) * N + 256 * k.tile + 64 * (k.sub >> 1) + 4 * r;
#pragma unroll
        for (int b = 0; b < 16; ++b) raw[b] = *(const unsigned*)(src + (size_t)b * N);
    }
}

__device__ __forceinline__ void task_tile(const Task& k, const unsigned (&raw)[16], u32x4 (&o)[4], unsigned (&off)[4]) {
    if (k.kind != 1) {
        const unsigned step = k.kind == 0 ? 1024u : (unsigned)(N * 4);
#pragma unroll
        for (int q = 0; q < 4; ++q) { o[q] = (u32x4){raw[4 * q], raw[4 * q + 1], raw[4 * q + 2], raw[4 * q + 3]}; off[q] = k.base + q * step; }
    } else {
#pragma unroll
        for (int a = 0; a < 4; ++a) {
            unsigned o0, o1, o2, o3;
            transpose4(raw[4 * a], raw[4 * a + 1], raw[4 * a + 2], raw[4 * a + 3], o0, o1, o2, o3);
            o[0][a] = o0; o[1][a] = o1; o[2][a] = o2; o[3][a] = o3;
        }
#pragma unroll
        for (int q = 0; q < 4; ++q) off[q] = k.base + (q >> 1) * HTB + (q & 1) * 2048;
    }
}

__device__ __forceinline__ void comm_tasks(const Params& p, LAS unsigned char* stage, int cw, int lane) {
    const unsigned call = (unsigned)__builtin_amdgcn_readfirstlane((int)load_epoch(p)) + 1u;
    unsigned* done = (unsigned*)(p.ws[NR - 1] + WS_DONE);
    unsigned* claim = (unsigned*)(p.ws[NR - 1] + WS_CLAIM);
    const unsigned* tcnt = (const unsigned*)(p.ws[NR - 1] + WS_TCNT);
    unsigned* tmo = (unsigned*)(p.ws[NR - 1] + WS_TMO);
    const unsigned cbase = (call - 1u) * (unsigned)NTASK;
    const unsigned par = (call & 1u) * (unsigned)IMG;
    const size_t otile = ((size_t)(8 * (p.rank & 1) * 256) * N + (NREG + (p.rank >> 1)) * 256) * 4;
    const int orec = 2048 * N * 4 - (NREG + (p.rank >> 1)) * 1024;
    const unsigned sbase = (unsigned)(size_t)stage;
    const int r = lane & 15, g = lane >> 4;
    int ready = -1;
#define FETCH_TASK(k) do { \
        const unsigned char* _s; size_t _d1, _d2; \
        if (k.kind == 0) { _s = p.x + (size_t)(256 * k.tile + 32 * k.sub + r) * KS + 128 * k.kk + 16 * g; _d1 = 64; _d2 = 16 * KS; } \
        else if (k.kind == 1) { _s = p.w + (size_t)(128 * k.kk + 64 * (k.sub & 1) + (lane >> 2)) * N + 256 * k.tile + 64 * (k.sub >> 1) + 16 * (lane & 3); \
            _d1 = (size_t)16 * N; _d2 = (size_t)32 * N; } \
        else { \
            if (k.tile > ready) { \
                const unsigned long long _t0 = __builtin_amdgcn_s_memrealtime(); \
                while ((int)(__hip_atomic_load(tcnt + k.tile * 32, __ATOMIC_RELAXED, __HIP_MEMORY_SCOPE_AGENT) - 8u * call) < 0) { \
                    if (__hip_atomic_load(tmo, __ATOMIC_RELAXED, __HIP_MEMORY_SCOPE_AGENT) == call || \
                        __builtin_amdgcn_s_memrealtime() - _t0 > DEADLINE_TICKS) { \
                        __hip_atomic_store(tmo, call, __ATOMIC_RELAXED, __HIP_MEMORY_SCOPE_AGENT); \
                        break; } \
                    __builtin_amdgcn_s_sleep(8); \
                } \
                __builtin_amdgcn_fence(__ATOMIC_ACQUIRE, "agent"); \
                __builtin_amdgcn_s_waitcnt(0x0070); \
                ready = k.tile; \
            } \
            _s = (const unsigned char*)p.out[NR - 1] + otile + k.base; _d1 = (size_t)N * 4; _d2 = (size_t)N * 8; \
        } \
        unsigned _m0; \
        asm volatile("s_mov_b32 %0, m0\n\ts_mov_b32 m0, %5\n\ts_nop 0\n\tglobal_load_lds_dwordx4 %1, off\n\t" \
                     "s_add_u32 m0, m0, 0x400\n\ts_nop 0\n\tglobal_load_lds_dwordx4 %2, off\n\t" \
                     "s_add_u32 m0, m0, 0x400\n\ts_nop 0\n\tglobal_load_lds_dwordx4 %3, off\n\t" \
                     "s_add_u32 m0, m0, 0x400\n\ts_nop 0\n\tglobal_load_lds_dwordx4 %4, off\n\ts_mov_b32 m0, %0" \
                     : "=&s"(_m0) : "v"(_s), "v"(_s + _d1), "v"(_s + _d2), "v"(_s + _d1 + _d2), "s"(sbase) : "memory", "scc"); } while (0)
    int prev = -1, cur = cw, nxt = cw + NCW;
    Task k;
    task_decode(p.rank, cur, lane, k);
    FETCH_TASK(k);
    asm volatile("s_waitcnt vmcnt(0)" ::: "memory");
    for (;;) {
        unsigned raw[16], off[4];
        u32x4 o[4];
        if (k.kind == 1) {
#pragma unroll
            for (int b = 0; b < 16; ++b) raw[b] = *(const LAS unsigned*)(stage + g * 1024 + b * 64 + r * 4);
        } else {
#pragma unroll
            for (int q = 0; q < 4; ++q) {
                const u32x4 v = *(const LAS u32x4*)(stage + q * 1024 + lane * 16);
                raw[4 * q] = v[0]; raw[4 * q + 1] = v[1]; raw[4 * q + 2] = v[2]; raw[4 * q + 3] = v[3];
            }
        }
        task_tile(k, raw, o, off);
        asm volatile("s_waitcnt lgkmcnt(0)" : "+v"(o[0]), "+v"(o[1]), "+v"(o[2]), "+v"(o[3]) :: "memory");
        const bool is_out = k.kind == 2;
        const unsigned add = is_out ? 0u : par;
        __amdgpu_buffer_rsrc_t rs[NR - 1];
#pragma unroll
        for (int d = 0; d < NR - 1; ++d) {
            const bool want = k.only < 0 || (((p.rank + 1 + d) & 7) >> 1) == k.only;
            rs[d] = __builtin_amdgcn_make_buffer_rsrc(is_out ? (unsigned char*)p.out[d] + otile : p.ws[d] + WS_IMG, 0,
                                                      want ? (is_out ? orec : (int)(2 * IMG)) : 0, 0x00020000);
        }
        const bool more = (unsigned)nxt < (unsigned)NTASK;
        task_decode(p.rank, more ? nxt : cur, lane, k);
        FETCH_TASK(k);
        unsigned tk = 1u; unsigned long long ex;
        asm volatile("s_mov_b64 %1, exec\n\ts_mov_b64 exec, 1\n\tglobal_atomic_add %0, %2, %0, off sc0\n\ts_mov_b64 exec, %1"
                     : "+v"(tk), "=&s"(ex) : "v"(claim) : "memory");
#pragma unroll
        for (int q = 0; q < 4; ++q)
#pragma unroll
            for (int d = 0; d < NR - 1; ++d) __builtin_amdgcn_raw_buffer_store_b128(o[q], rs[d], off[q] + add, 0, 17);
        asm volatile("s_waitcnt vmcnt(28)" : "+v"(tk) :: "memory");
        const int got = 2 * NCW + (int)((unsigned)__builtin_amdgcn_readfirstlane((int)tk) - cbase);
        if (prev >= 0 && lane == 0) __hip_atomic_store(done + prev, call, __ATOMIC_RELAXED, __HIP_MEMORY_SCOPE_AGENT);
        prev = cur;
        if (!more) break;
        cur = nxt;
        nxt = got;
    }
    asm volatile("s_waitcnt vmcnt(0)" ::: "memory");
    if (lane == 0) __hip_atomic_store(done + prev, call, __ATOMIC_RELAXED, __HIP_MEMORY_SCOPE_AGENT);
    STAMP(p, 12 + (cw & 3));
#undef FETCH_TASK
}

__device__ __forceinline__ void comm_publish(const Params& p, int lane) {
    const unsigned call = (unsigned)__builtin_amdgcn_readfirstlane((int)load_epoch(p)) + 1u;
    const unsigned* done = (const unsigned*)(p.ws[NR - 1] + WS_DONE);
    unsigned* tmo = (unsigned*)(p.ws[NR - 1] + WS_TMO);
    unsigned* dst = (unsigned*)(p.ws[0] + WS_PROG);
#pragma unroll
    for (int d = 1; d < NR - 1; ++d) if (lane == d) dst = (unsigned*)(p.ws[d] + WS_PROG);
    dst += p.rank;
    const unsigned base = (call - 1u) * NBLK;
    const unsigned long long t0 = __builtin_amdgcn_s_memrealtime();
    int nt = 0, pub = 0;
    bool dead = false;
    while (nt < NTASK) {
        const int idx = nt + lane;
        unsigned v = 0;
        if (idx < NTASK) v = __hip_atomic_load(done + idx, __ATOMIC_RELAXED, __HIP_MEMORY_SCOPE_AGENT);
        const unsigned long long miss = ~__ballot(v == call);
        const int n = miss ? __builtin_ctzll(miss) : 64;
        nt += n;
        if ((nt >> 3) > pub) {
            pub = nt >> 3;
            if (lane < NR - 1) __hip_atomic_store(dst, base + (unsigned)pub, __ATOMIC_RELAXED, __HIP_MEMORY_SCOPE_SYSTEM);
        }
        if (n == 0) {
            if (__builtin_amdgcn_s_memrealtime() - t0 > DEADLINE_TICKS) { dead = true; break; }
            __builtin_amdgcn_s_sleep(2);
        }
    }
    const unsigned* progl = (const unsigned*)(p.ws[NR - 1] + WS_PROG) + (lane & 7);
    while (!dead) {
        const unsigned v = __hip_atomic_load(progl, __ATOMIC_RELAXED, __HIP_MEMORY_SCOPE_SYSTEM);
        if (__all((int)(v - (base + NBLK)) >= 0)) break;
        if (__builtin_amdgcn_s_memrealtime() - t0 > DEADLINE_TICKS) dead = true;
        __builtin_amdgcn_s_sleep(2);
    }
    __builtin_amdgcn_fence(__ATOMIC_ACQUIRE, "");
    if (dead || __hip_atomic_load(tmo, __ATOMIC_RELAXED, __HIP_MEMORY_SCOPE_AGENT) == call) {
        if (lane == 0) {
            __hip_atomic_store(tmo, call, __ATOMIC_RELAXED, __HIP_MEMORY_SCOPE_AGENT);
            p.out[NR - 1][(size_t)(8 * (p.rank & 1) * 256) * N + (NREG + (p.rank >> 1)) * 256] = __builtin_nanf("");
        }
    }
    STAMP(p, 15);
}

struct Tile { int i, j, need0, step, sh; };

__device__ __forceinline__ void tile_of(int rank, int xcd, int e, Tile& t) {
    if (xcd >= 6) {
        if (e < 4) {
            t.sh = (xcd - 6) * 4 + e;
            t.i = 8 * (rank & 1) + t.sh; t.j = NREG + (rank >> 1);
            t.need0 = 5 + 2 * t.i; t.step = 36;
            return;
        }
        e -= 4;
    }
    t.sh = -1;
    t.i = 4 * (xcd & 3) + (e & 3);
    t.j = 2 * (e >> 2) + (xcd >> 2);
    if (t.j < 16) { t.need0 = 5 + (2 * t.i > 2 * t.j + 1 ? 2 * t.i : 2 * t.j + 1); t.step = 36; }
    else if (t.j < 48) { t.need0 = 145 + ((t.j >> 4) - 1) * 64 + (t.j & 15); t.step = 16; }
    else { t.need0 = 273 + (t.j - 48); t.step = 12; }
}

__device__ __forceinline__ void gemm_role(const Params& p, LAS unsigned char* lds, int xcd, int rk) {
    const unsigned call = (unsigned)__builtin_amdgcn_readfirstlane((int)load_epoch(p)) + 1u;
    const int tid = threadIdx.x, wid = __builtin_amdgcn_readfirstlane(tid >> 6), lane = tid & 63;
    const int wr = wid >> 2, wc = wid & 3;
    const unsigned char* aimg = p.ws[NR - 1] + WS_IMG + (size_t)(call & 1u) * IMG;
    const unsigned char* bimg = aimg + AIMG;
    const unsigned* prog = (const unsigned*)(p.ws[NR - 1] + WS_PROG);
    unsigned* tmo = (unsigned*)(p.ws[NR - 1] + WS_TMO);
    LAS volatile unsigned* ctrl = (LAS volatile unsigned*)(lds + LDS_CTRL);
    const unsigned pbase = (call - 1u) * NBLK;
    unsigned voff0 = (unsigned)tid * 16u, voff1 = voff0 + 8192u;
    asm volatile("" : "+v"(voff0), "+v"(voff1));
    const unsigned ldsw = (unsigned)wid * 1024u;
    int aoff = wr * 8192 + lane * 16, boff = 4 * HTB + wc * 4096 + lane * 16;
    asm volatile("" : "+v"(aoff), "+v"(boff));
    const int rank = p.rank, nq = xcd < 6 ? 30 : 31;
    unsigned vprog = pbase;
    const unsigned* progl = prog + (lane & 7);

#define SA(b, h) (((b) * 2 + (h)) * HTB)
#define SB(b, h) ((4 + (b) * 2 + (h)) * HTB)
#define STAGE(bufoff, gbase) do { const unsigned char* _g = (gbase); \
        __builtin_amdgcn_global_load_lds((const unsigned*)(_g + voff0), (LAS unsigned*)(lds + (bufoff) + ldsw), 16, 0, 0); \
        __builtin_amdgcn_global_load_lds((const unsigned*)(_g + voff1), (LAS unsigned*)(lds + (bufoff) + ldsw + 8192), 16, 0, 0); } while (0)
#define FRAG(off) __builtin_shufflevector(*(const LAS i32x4*)(lds + (off)), *(const LAS i32x4*)(lds + (off) + 1024), 0, 1, 2, 3, 4, 5, 6, 7)
#define LDA(dst, b, h) do { _Pragma("unroll") for (int m = 0; m < 4; ++m) dst[m] = FRAG(SA(b, h) + aoff + m * 2048); } while (0)
#define LDB(dst, b, h) do { _Pragma("unroll") for (int n = 0; n < 2; ++n) dst[n] = FRAG(SA(b, h) + boff + n * 2048); } while (0)
#define MMA(ai, bj, At, Bt, Z) do { __builtin_amdgcn_s_setprio(1); _Pragma("unroll") for (int m = 0; m < 4; ++m) _Pragma("unroll") for (int n = 0; n < 2; ++n) \
        acc[ai][bj][m][n] = __builtin_amdgcn_mfma_scale_f32_16x16x128_f8f6f4(At[m], Bt[n], (Z) ? (f32x4){0.f, 0.f, 0.f, 0.f} : acc[ai][bj][m][n], 0, 0, 0, 0, 0, 0); \
        __builtin_amdgcn_s_setprio(0); } while (0)
#define WAIT_V(n) asm volatile("s_waitcnt vmcnt(" #n ")" ::: "memory")
#define WAIT_L(n) asm volatile("s_waitcnt lgkmcnt(" #n ")" ::: "memory")
#define BAR __builtin_amdgcn_s_barrier()
#define SCHED __builtin_amdgcn_sched_barrier(0)
#define KOFF(t) ((size_t)((((rank + (t)) & 7) * 4 + ((t) >> 3)) * BLK))
#define GATE(need0, step, t) do { if (GATE_ON && wid == 0) { \
        const int _src = (rank + (t)) & 7; \
        const unsigned _need = pbase + (unsigned)((need0) + ((t) >> 3) * (step)); \
        if ((int)((unsigned)__builtin_amdgcn_readlane((int)vprog, _src) - _need) < 0) { \
            const unsigned long long _t0 = __builtin_amdgcn_s_memrealtime(); \
            for (;;) { \
                vprog = __hip_atomic_load(progl, __ATOMIC_RELAXED, __HIP_MEMORY_SCOPE_SYSTEM); \
                if ((int)((unsigned)__builtin_amdgcn_readlane((int)vprog, _src) - _need) >= 0) break; \
                if (__hip_atomic_load(tmo, __ATOMIC_RELAXED, __HIP_MEMORY_SCOPE_AGENT) == call || \
                    __builtin_amdgcn_s_memrealtime() - _t0 > DEADLINE_TICKS) { \
                    __hip_atomic_store(tmo, call, __ATOMIC_RELAXED, __HIP_MEMORY_SCOPE_AGENT); \
                    ctrl[0] = 1u; break; } \
                __builtin_amdgcn_s_sleep(4); \
            } \
            __builtin_amdgcn_s_waitcnt(0xC07F); \
            asm volatile("" ::: "memory"); \
        } } } while (0)
#define POLL_ISSUE() do { if (wid == 0) asm volatile("global_load_dword %0, %1, off sc0 sc1" : "+v"(vprog) : "v"(progl) : "memory"); } while (0)
#define POLL_LANDED() asm volatile("" : "+v"(vprog))

#define ST8(ai, mh, AUX) { float _sc = osc; int _vo = vo; asm volatile("" : "+v"(_sc), "+v"(_vo)); \
        _Pragma("unroll") for (int m = 2 * (mh); m < 2 * (mh) + 2; ++m) _Pragma("unroll") for (int e = 0; e < 4; ++e) { \
            const f32x4 v = {acc[ai][0][m][0][e] * _sc, acc[ai][0][m][1][e] * _sc, acc[ai][1][m][0][e] * _sc, acc[ai][1][m][1][e] * _sc}; \
            __builtin_amdgcn_raw_buffer_store_b128(__builtin_bit_cast(u32x4, v), ro, _vo + ((ai) * 128 + m * 16 + e) * (N * 4), 0, AUX); } }
#define STORES(ai, mh) do { if (osh < 0) ST8(ai, mh, OUT_AUX) else ST8(ai, mh, 16) } while (0)
#define ITER(FIRST, LAST) do { \
        const int t2 = (t + 2) & (NKS - 1), t3 = t2 + 1; \
        const unsigned char* a1 = cA + KOFF(t + 1); \
        const unsigned char* a2 = ((LAST) ? nA : cA) + KOFF(t2); \
        const unsigned char* b2 = ((LAST) ? nB : cB) + KOFF(t2); \
        const unsigned char* a3 = ((LAST) ? nA : cA) + KOFF(t3); \
        const unsigned char* b3 = ((LAST) ? nB : cB) + KOFF(t3); \
        GATE((LAST) ? nxt.need0 : cur.need0, (LAST) ? nxt.step : cur.step, t2); \
        POLL_ISSUE(); \
        LDB(B0, 0, 0); SCHED; LDA(At, 0, 0); STAGE(SA(1, 1), a1 + HTB); \
        if (FIRST) STORES(1, 0); \
        WAIT_L(8); BAR; WAIT_L(0); MMA(0, 0, At, B0, FIRST); BAR; SCHED; \
        LDB(B1, 0, 1); STAGE(SB(0, 0), b2); \
        if (FIRST) STORES(1, 1); \
        BAR; WAIT_L(0); MMA(0, 1, At, B1, FIRST); BAR; \
        LDA(At, 0, 1); STAGE(SA(0, 0), a2); \
        BAR; WAIT_L(0); MMA(1, 0, At, B0, FIRST); BAR; SCHED; \
        STAGE(SB(0, 1), b2 + HTB); \
        if (FIRST) WAIT_V(22); else WAIT_V(6); \
        POLL_LANDED(); BAR; MMA(1, 1, At, B1, FIRST); BAR; \
        GATE((LAST) ? nxt.need0 : cur.need0, (LAST) ? nxt.step : cur.step, t3); \
        POLL_ISSUE(); \
        unsigned dead = 0u; \
        if (LAST) dead = ctrl[0]; \
        LDB(B0, 1, 0); SCHED; LDA(At, 1, 0); STAGE(SA(0, 1), a2 + HTB); \
        WAIT_L(8); BAR; WAIT_L(0); MMA(0, 0, At, B0, 0); BAR; SCHED; \
        LDB(B1, 1, 1); STAGE(SB(1, 0), b3); \
        BAR; WAIT_L(0); MMA(0, 1, At, B1, 0); BAR; \
        LDA(At, 1, 1); STAGE(SA(1, 0), a3); \
        if (LAST) { \
            ro = __builtin_amdgcn_make_buffer_rsrc(p.out[NR - 1] + (size_t)(cur.i * 256 + wr * 64) * N + cur.j * 256 + wc * 64, 0, 192 * N * 4, 0x00020000); \
            osh = cur.sh; osc = dead ? __builtin_nanf("") : scale; \
            STORES(0, 0); \
        } \
        BAR; WAIT_L(0); MMA(1, 0, At, B0, 0); BAR; SCHED; \
        STAGE(SB(1, 1), b3 + HTB); \
        if (LAST) WAIT_V(14); else WAIT_V(6); \
        POLL_LANDED(); \
        if (LAST) STORES(0, 1); \
        if (FIRST) { if (osh >= 0 && lane == 0) __hip_atomic_fetch_add(tcnt + osh * 32, 1u, __ATOMIC_RELAXED, __HIP_MEMORY_SCOPE_AGENT); } \
        BAR; MMA(1, 1, At, B1, 0); BAR; \
    } while (0)

    Tile cur, nxt;
    tile_of(rank, xcd, rk, cur);
    f32x4 acc[2][2][4][2];
#pragma unroll
    for (int b = 0; b < 2; ++b)
#pragma unroll
        for (int m = 0; m < 4; ++m)
#pragma unroll
            for (int n = 0; n < 2; ++n) acc[1][b][m][n] = (f32x4){0.f, 0.f, 0.f, 0.f};
    i32x8 At[4], B0[2], B1[2];
    const unsigned char* cA = aimg + (size_t)cur.i * NKS * BLK;
    const unsigned char* cB = bimg + (size_t)cur.j * NKS * BLK;
    const float scale = p.sx[0] * p.sw[0];
    unsigned* tcnt = (unsigned*)(p.ws[NR - 1] + WS_TCNT);
    __amdgpu_buffer_rsrc_t ro = __builtin_amdgcn_make_buffer_rsrc(p.out[NR - 1], 0, 0, 0x00020000);
    int osh = -1;
    float osc = scale;
    const int vo = (lane >> 4) * (4 * N * 4) + (lane & 15) * 16;

    {
        const __amdgpu_buffer_rsrc_t rl = __builtin_amdgcn_make_buffer_rsrc((void*)aimg, 0, (int)IMG, 0x00020000);
        for (int t = ((int)blockIdx.x - NCOMM) * 8 + wid; t < NTASK_IN; t += (NWG - NCOMM) * 8) {
            unsigned raw[16], off[4];
            Task k; u32x4 o[4];
            task_decode(rank, t, lane, k);
            task_load(p, k, lane, raw);
            task_tile(k, raw, o, off);
            if (k.only < 0 || k.only == (rank >> 1)) {
#pragma unroll
                for (int q = 0; q < 4; ++q) __builtin_amdgcn_raw_buffer_store_b128(o[q], rl, off[q], 0, 16);
            }
        }
        WAIT_V(0);
        BAR;
        if (tid == 0) {
            const unsigned n = __hip_atomic_fetch_add((unsigned*)(p.ws[NR - 1] + WS_LCNT), 1u, __ATOMIC_RELAXED, __HIP_MEMORY_SCOPE_AGENT) + 1u;
            if (n == call * (unsigned)(NWG - NCOMM))
                __hip_atomic_store((unsigned*)(p.ws[NR - 1] + WS_PROG) + rank, pbase + NBLK, __ATOMIC_RELAXED, __HIP_MEMORY_SCOPE_SYSTEM);
            ctrl[0] = 0u;
            __builtin_amdgcn_fence(__ATOMIC_ACQUIRE, "");
            asm volatile("s_waitcnt vmcnt(0) lgkmcnt(0)" ::: "memory");
        }
    }
    GATE(cur.need0, cur.step, 0); GATE(cur.need0, cur.step, 1);
    BAR;
    STAGE(SB(0, 0), cB + KOFF(0)); STAGE(SA(0, 0), cA + KOFF(0)); STAGE(SB(0, 1), cB + KOFF(0) + HTB); STAGE(SA(0, 1), cA + KOFF(0) + HTB);
    if (wr == 1) BAR;
    WAIT_V(4); BAR;
    STAGE(SB(1, 0), cB + KOFF(1)); STAGE(SA(1, 0), cA + KOFF(1)); STAGE(SB(1, 1), cB + KOFF(1) + HTB);
    WAIT_V(6); BAR;
    if (wid == 0) STAMP(p, 1);
#pragma unroll 1
    for (int un = 0; un < 4; ++un) {
        tile_of(rank, xcd, rk + (un < 3 ? un + 1 : un) * nq, nxt);
        const unsigned char* nA = aimg + (size_t)nxt.i * NKS * BLK;
        const unsigned char* nB = bimg + (size_t)nxt.j * NKS * BLK;
        int t = 0;
        ITER(1, 0);
#pragma unroll 1
        for (t = 2; t < NKS - 2; t += 2) ITER(0, 0);
        ITER(0, 1);
        if (wid == 0) STAMP(p, 2 + 2 * un);
        cur = nxt; cA = nA; cB = nB;
    }
    STORES(1, 0);
    STORES(1, 1);
    if (wid == 0) STAMP(p, 9);
    WAIT_V(0);
    if (wr == 0) BAR;
#undef ST8
#undef STORES
#undef ITER
#undef SA
#undef SB
#undef STAGE
#undef FRAG
#undef LDA
#undef LDB
#undef MMA
#undef WAIT_V
#undef WAIT_L
#undef BAR
#undef SCHED
#undef KOFF
#undef GATE
#undef POLL_ISSUE
#undef POLL_LANDED
}

__global__ void __launch_bounds__(512, 2) fused_gemm_allgather(const Params p) {
    extern __shared__ __attribute__((aligned(16))) unsigned char shm[];
    LAS unsigned char* lds = (LAS unsigned char*)shm;
    const int b = blockIdx.x;
    if (threadIdx.x == 0) STAMP(p, 0);
    if (b < NCOMM) {
        const int wid = __builtin_amdgcn_readfirstlane((int)(threadIdx.x >> 6)), lane = threadIdx.x & 63;
        bool run = true;
        if (run) {
            const int cw = b == 0 ? (wid >= 1 && wid <= TW0 ? wid - 1 : -1) : (wid < TW ? TW0 + (b - 1) * TW + wid : -1);
            if (cw >= 0) comm_tasks(p, lds + wid * 4096, cw, lane);
            else if (b == 0 && wid == 0) comm_publish(p, lane);
        }
    } else {
        const int xcd = b & 7;
        gemm_role(p, lds, xcd, (b >> 3) - (xcd < 6 ? 2 : 1));
    }
    __builtin_amdgcn_s_barrier();
    if (threadIdx.x == 0) {
        const unsigned ticket = __hip_atomic_fetch_add((unsigned*)(p.ws[NR - 1] + WS_TICKET), 1u, __ATOMIC_RELAXED, __HIP_MEMORY_SCOPE_AGENT) + 1u;
        if (ticket % NWG == 0u) __hip_atomic_store((unsigned*)(p.ws[NR - 1] + WS_EPOCH), ticket / NWG, __ATOMIC_RELAXED, __HIP_MEMORY_SCOPE_AGENT);
    }
}

}

extern "C" void kernel_launch(int rank, int nranks, ncclComm_t comm,
                              hipStream_t stream,
                              void* const* d_in, const int* in_sizes, int n_in,
                              void* d_out, int out_size,
                              const int* in_dtypes, int out_dtype,
                              void* const* d_out_all,
                              void* const* d_ws_all, size_t ws_bytes) {
    (void)comm; (void)in_sizes; (void)in_dtypes; (void)out_dtype; (void)d_out;
    if (nranks != NR || n_in != 4 || (unsigned)rank >= (unsigned)NR || ws_bytes < WS_NEED || out_size != M * N) {
        fprintf(stderr, "kernel_launch: unexpected problem shape\n");
        return;
    }
    Params p;
    p.x = (const unsigned char*)d_in[0];
    p.w = (const unsigned char*)d_in[1];
    p.sx = (const float*)d_in[2];
    p.sw = (const float*)d_in[3];
    for (int i = 0; i < NR; ++i) {
        p.ws[i] = (unsigned char*)d_ws_all[(rank + 1 + i) % NR];
        p.out[i] = (float*)d_out_all[(rank + 1 + i) % NR];
    }
    p.rank = rank;
    HIP_CHECK(hipFuncSetAttribute((const void*)fused_gemm_allgather, hipFuncAttributeMaxDynamicSharedMemorySize, LDS_BYTES));
    hipLaunchKernelGGL(fused_gemm_allgather, dim3(NWG), dim3(512), LDS_BYTES, stream, p);
    HIP_CHECK(hipGetLastError());
}
```

```cpp
#include <hip/hip_runtime.h>
#include <rccl/rccl.h>
#include <cstdio>
#include <cstdint>

#define HIP_CHECK(expr) do { hipError_t _e = (expr); if (_e != hipSuccess) { \
    fprintf(stderr, "HIP error %s at %s:%d\n", hipGetErrorString(_e), __FILE__, __LINE__); \
    return; } } while (0)

namespace {

#define LAS __attribute__((address_space(3)))
typedef int i32x4 __attribute__((ext_vector_type(4)));
typedef int i32x8 __attribute__((ext_vector_type(8)));
typedef unsigned u32x4 __attribute__((ext_vector_type(4)));
typedef float f32x4 __attribute__((ext_vector_type(4)));

constexpr int M = 4096, N = 16384, NR = 8, KS = 512;
constexpr int NKS = 32;
constexpr int BLK = 32768;
constexpr int HTB = 16384;
constexpr int NMT = 16, NNT = 64, NREG = 60;
constexpr size_t AIMG = (size_t)NMT * NKS * BLK, BIMG = (size_t)NNT * NKS * BLK, IMG = AIMG + BIMG;

constexpr int NBLK_IN = 320, NBLK = 384;
constexpr int NTASK_IN = NBLK_IN * 8, NTASK = NBLK * 8;
#define TW 2
#define TW0 2
#define SNAKE 1
#define OUT_AUX 2
constexpr int NWG = 256, NCOMM = 14;
constexpr int NCW = (NCOMM - 1) * TW + TW0;

constexpr size_t WS_TICKET = 0, WS_EPOCH = 128, WS_TMO = 256, WS_PROG = 512, WS_CLAIM = 768, WS_LCNT = 896, WS_TCNT = 1024, WS_DONE = 4096, WS_IMG = 1u << 20;
constexpr size_t WS_STAMP = 65536;
constexpr size_t WS_NEED = WS_IMG + 2 * IMG;
#define STAMP(p, k) do { } while (0)
constexpr unsigned long long DEADLINE_TICKS = 50000000ull;

constexpr int LDS_CTRL = 8 * HTB;
constexpr int LDS_BYTES = LDS_CTRL + 64;

struct Params {
    const unsigned char* x;
    const unsigned char* w;
    const float* sx;
    const float* sw;
    unsigned char* ws[NR];
    float* out[NR];
    int rank;
};

#define GATE_ON 1

__device__ __forceinline__ void transpose4(unsigned d0, unsigned d1, unsigned d2, unsigned d3,
                                           unsigned& o0, unsigned& o1, unsigned& o2, unsigned& o3) {
    const unsigned t0 = __builtin_amdgcn_perm(d1, d0, 0x05010400u), t1 = __builtin_amdgcn_perm(d1, d0, 0x07030602u);
    const unsigned t2 = __builtin_amdgcn_perm(d3, d2, 0x05010400u), t3 = __builtin_amdgcn_perm(d3, d2, 0x07030602u);
    o0 = __builtin_amdgcn_perm(t2, t0, 0x05040100u); o1 = __builtin_amdgcn_perm(t2, t0, 0x07060302u);
    o2 = __builtin_amdgcn_perm(t3, t1, 0x05040100u); o3 = __builtin_amdgcn_perm(t3, t1, 0x07060302u);
}

__device__ __forceinline__ unsigned load_epoch(const Params& p) {
    return __hip_atomic_load((const unsigned*)(p.ws[NR - 1] + WS_EPOCH), __ATOMIC_RELAXED, __HIP_MEMORY_SCOPE_AGENT);
}

struct Task { int kind, only, tile, kk, sub; unsigned base; };

__device__ __forceinline__ void task_decode(int rank, int t, int lane, Task& k) {
    const int bp = t >> 3;
    k.sub = t & 7; k.only = -1; k.kind = 1;
    if (bp < 128) { k.kk = bp >> 5; k.kind = bp & 1; k.tile = (bp & 31) >> 1; }
    else if (bp < 208) {
        k.kk = (bp - 128) / 20; const int q = bp - 128 - k.kk * 20;
        if (q < 4) { k.tile = NREG + q; k.only = q; } else k.tile = 12 + q;
    } else if (bp < 272) { const int q = bp - 208; k.kk = q >> 4; k.tile = 32 + (q & 15); }
    else if (bp < NBLK_IN) { const int q = bp - 272; k.kk = q / 12; k.tile = 48 + (q - k.kk * 12); }
    else { k.kind = 2; k.kk = 0; k.tile = (bp - NBLK_IN) >> 3; }
    const unsigned blk = (unsigned)(k.tile * NKS + rank * 4 + k.kk) * BLK + lane * 16;
    k.base = k.kind == 0 ? blk + k.sub * 4096
           : k.kind == 1 ? (unsigned)AIMG + blk + (k.sub >> 1) * 4096 + (k.sub & 1) * 1024
                         : (unsigned)((t - NTASK_IN) * 4) * (unsigned)(N * 4) + lane * 16;
}

__device__ __forceinline__ void task_load(const Params& p, const Task& k, int lane, unsigned (&raw)[16]) {
    const int r = lane & 15, g = lane >> 4;
    if (k.kind == 0) {
        const unsigned char* src = p.x + (size_t)(256 * k.tile + 32 * k.sub + r) * KS + 128 * k.kk + 16 * g;
#pragma unroll
        for (int q = 0; q < 4; ++q) {
            const u32x4 v = *(const u32x4*)(src + (q >> 1) * 16 * KS + (q & 1) * 64);
            raw[4 * q] = v[0]; raw[4 * q + 1] = v[1]; raw[4 * q + 2] = v[2]; raw[4 * q + 3] = v[3];
        }
    } else {
        const unsigned char* src = p.w + (size_t)(128 * k.kk + 64 * (k.sub & 1) + 16 * g) * N + 256 * k.tile + 64 * (k.sub >> 1) + 4 * r;
#pragma unroll
        for (int b = 0; b < 16; ++b) raw[b] = *(const unsigned*)(src + (size_t)b * N);
    }
}

__device__ __forceinline__ void task_tile(const Task& k, const unsigned (&raw)[16], u32x4 (&o)[4], unsigned (&off)[4]) {
    if (k.kind != 1) {
        const unsigned step = k.kind == 0 ? 1024u : (unsigned)(N * 4);
#pragma unroll
        for (int q = 0; q < 4; ++q) { o[q] = (u32x4){raw[4 * q], raw[4 * q + 1], raw[4 * q + 2], raw[4 * q + 3]}; off[q] = k.base + q * step; }
    } else {
#pragma unroll
        for (int a = 0; a < 4; ++a) {
            unsigned o0, o1, o2, o3;
            transpose4(raw[4 * a], raw[4 * a + 1], raw[4 * a + 2], raw[4 * a + 3], o0, o1, o2, o3);
            o[0][a] = o0; o[1][a] = o1; o[2][a] = o2; o[3][a] = o3;
        }
#pragma unroll
        for (int q = 0; q < 4; ++q) off[q] = k.base + (q >> 1) * HTB + (q & 1) * 2048;
    }
}

__device__ __forceinline__ void comm_tasks(const Params& p, LAS unsigned char* stage, int cw, int lane) {
    unsigned epoch = load_epoch(p), call = 0u;
    unsigned* done = (unsigned*)(p.ws[NR - 1] + WS_DONE);
    unsigned* claim = (unsigned*)(p.ws[NR - 1] + WS_CLAIM);
    const unsigned* tcnt = (const unsigned*)(p.ws[NR - 1] + WS_TCNT);
    unsigned* tmo = (unsigned*)(p.ws[NR - 1] + WS_TMO);
    const size_t otile = ((size_t)(8 * (p.rank & 1) * 256) * N + (NREG + (p.rank >> 1)) * 256) * 4;
    const int orec = 2048 * N * 4 - (NREG + (p.rank >> 1)) * 1024;
    const unsigned sbase = (unsigned)(size_t)stage;
    const int r = lane & 15, g = lane >> 4;
    int ready = -1;
#define FETCH_TASK(k) do { \
        const unsigned char* _s; size_t _d1, _d2; \
        if (k.kind == 0) { _s = p.x + (size_t)(256 * k.tile + 32 * k.sub + r) * KS + 128 * k.kk + 16 * g; _d1 = 64; _d2 = 16 * KS; } \
        else if (k.kind == 1) { _s = p.w + (size_t)(128 * k.kk + 64 * (k.sub & 1) + (lane >> 2)) * N + 256 * k.tile + 64 * (k.sub >> 1) + 16 * (lane & 3); \
            _d1 = (size_t)16 * N; _d2 = (size_t)32 * N; } \
        else { \
            if (k.tile > ready) { \
                const unsigned long long _t0 = __builtin_amdgcn_s_memrealtime(); \
                while ((int)(__hip_atomic_load(tcnt + k.tile * 32, __ATOMIC_RELAXED, __HIP_MEMORY_SCOPE_AGENT) - 8u * call) < 0) { \
                    if (__hip_atomic_load(tmo, __ATOMIC_RELAXED, __HIP_MEMORY_SCOPE_AGENT) == call || \
                        __builtin_amdgcn_s_memrealtime() - _t0 > DEADLINE_TICKS) { \
                        __hip_atomic_store(tmo, call, __ATOMIC_RELAXED, __HIP_MEMORY_SCOPE_AGENT); \
                        break; } \
                    __builtin_amdgcn_s_sleep(8); \
                } \
                __builtin_amdgcn_fence(__ATOMIC_ACQUIRE, "agent"); \
                __builtin_amdgcn_s_waitcnt(0x0070); \
                ready = k.tile; \
            } \
            _s = (const unsigned char*)p.out[NR - 1] + otile + k.base; _d1 = (size_t)N * 4; _d2 = (size_t)N * 8; \
        } \
        unsigned _m0; \
        asm volatile("s_mov_b32 %0, m0\n\ts_mov_b32 m0, %5\n\ts_nop 0\n\tglobal_load_lds_dwordx4 %1, off\n\t" \
                     "s_add_u32 m0, m0, 0x400\n\ts_nop 0\n\tglobal_load_lds_dwordx4 %2, off\n\t" \
                     "s_add_u32 m0, m0, 0x400\n\ts_nop 0\n\tglobal_load_lds_dwordx4 %3, off\n\t" \
                     "s_add_u32 m0, m0, 0x400\n\ts_nop 0\n\tglobal_load_lds_dwordx4 %4, off\n\ts_mov_b32 m0, %0" \
                     : "=&s"(_m0) : "v"(_s), "v"(_s + _d1), "v"(_s + _d2), "v"(_s + _d1 + _d2), "s"(sbase) : "memory", "scc"); } while (0)
    int prev = -1, cur = cw, nxt = cw + NCW;
    Task k;
    task_decode(p.rank, cur, lane, k);
    FETCH_TASK(k);
    asm volatile("s_waitcnt vmcnt(0)" : "+v"(epoch) :: "memory");
    call = (unsigned)__builtin_amdgcn_readfirstlane((int)epoch) + 1u;
    const unsigned cbase = (call - 1u) * (unsigned)NTASK;
    const unsigned par = (call & 1u) * (unsigned)IMG;
    for (;;) {
        unsigned raw[16], off[4];
        u32x4 o[4];
        if (k.kind == 1) {
#pragma unroll
            for (int b = 0; b < 16; ++b) raw[b] = *(const LAS unsigned*)(stage + g * 1024 + b * 64 + r * 4);
        } else {
#pragma unroll
            for (int q = 0; q < 4; ++q) {
                const u32x4 v = *(const LAS u32x4*)(stage + q * 1024 + lane * 16);
                raw[4 * q] = v[0]; raw[4 * q + 1] = v[1]; raw[4 * q + 2] = v[2]; raw[4 * q + 3] = v[3];
            }
        }
        task_tile(k, raw, o, off);
        asm volatile("s_waitcnt lgkmcnt(0)" : "+v"(o[0]), "+v"(o[1]), "+v"(o[2]), "+v"(o[3]) :: "memory");
        const bool is_out = k.kind == 2;
        const unsigned add = is_out ? 0u : par;
        __amdgpu_buffer_rsrc_t rs[NR - 1];
#pragma unroll
        for (int d = 0; d < NR - 1; ++d) {
            const bool want = k.only < 0 || (((p.rank + 1 + d) & 7) >> 1) == k.only;
            rs[d] = __builtin_amdgcn_make_buffer_rsrc(is_out ? (unsigned char*)p.out[d] + otile : p.ws[d] + WS_IMG, 0,
                                                      want ? (is_out ? orec : (int)(2 * IMG)) : 0, 0x00020000);
        }
        const bool more = (unsigned)nxt < (unsigned)NTASK;
        task_decode(p.rank, more ? nxt : cur, lane, k);
        FETCH_TASK(k);
        unsigned tk = 1u; unsigned long long ex;
        asm volatile("s_mov_b64 %1, exec\n\ts_mov_b64 exec, 1\n\tglobal_atomic_add %0, %2, %0, off sc0\n\ts_mov_b64 exec, %1"
                     : "+v"(tk), "=&s"(ex) : "v"(claim) : "memory");
#pragma unroll
        for (int q = 0; q < 4; ++q)
#pragma unroll
            for (int d = 0; d < NR - 1; ++d) __builtin_amdgcn_raw_buffer_store_b128(o[q], rs[d], off[q] + add, 0, 17);
        asm volatile("s_waitcnt vmcnt(28)" : "+v"(tk) :: "memory");
        const int got = 2 * NCW + (int)((unsigned)__builtin_amdgcn_readfirstlane((int)tk) - cbase);
        if (prev >= 0 && lane == 0) __hip_atomic_store(done + prev, call, __ATOMIC_RELAXED, __HIP_MEMORY_SCOPE_AGENT);
        prev = cur;
        if (!more) break;
        cur = nxt;
        nxt = got;
    }
    asm volatile("s_waitcnt vmcnt(0)" ::: "memory");
    if (lane == 0) __hip_atomic_store(done + prev, call, __ATOMIC_RELAXED, __HIP_MEMORY_SCOPE_AGENT);
    STAMP(p, 12 + (cw & 3));
#undef FETCH_TASK
}

__device__ __forceinline__ void comm_publish(const Params& p, int lane) {
    const unsigned call = (unsigned)__builtin_amdgcn_readfirstlane((int)load_epoch(p)) + 1u;
    const unsigned* done = (const unsigned*)(p.ws[NR - 1] + WS_DONE);
    unsigned* tmo = (unsigned*)(p.ws[NR - 1] + WS_TMO);
    unsigned* dst = (unsigned*)(p.ws[0] + WS_PROG);
#pragma unroll
    for (int d = 1; d < NR - 1; ++d) if (lane == d) dst = (unsigned*)(p.ws[d] + WS_PROG);
    dst += p.rank;
    const unsigned base = (call - 1u) * NBLK;
    const unsigned long long t0 = __builtin_amdgcn_s_memrealtime();
    int nt = 0, pub = 0;
    bool dead = false;
    while (nt < NTASK) {
        const int idx = nt + lane;
        unsigned v = 0;
        if (idx < NTASK) v = __hip_atomic_load(done + idx, __ATOMIC_RELAXED, __HIP_MEMORY_SCOPE_AGENT);
        const unsigned long long miss = ~__ballot(v == call);
        const int n = miss ? __builtin_ctzll(miss) : 64;
        nt += n;
        if ((nt >> 3) > pub) {
            pub = nt >> 3;
            if (lane < NR - 1) __hip_atomic_store(dst, base + (unsigned)pub, __ATOMIC_RELAXED, __HIP_MEMORY_SCOPE_SYSTEM);
        }
        if (n == 0) {
            if (__builtin_amdgcn_s_memrealtime() - t0 > DEADLINE_TICKS) { dead = true; break; }
            __builtin_amdgcn_s_sleep(2);
        }
    }
    const unsigned* progl = (const unsigned*)(p.ws[NR - 1] + WS_PROG) + (lane & 7);
    while (!dead) {
        const unsigned v = __hip_atomic_load(progl, __ATOMIC_RELAXED, __HIP_MEMORY_SCOPE_SYSTEM);
        if (__all((int)(v - (base + NBLK)) >= 0)) break;
        if (__builtin_amdgcn_s_memrealtime() - t0 > DEADLINE_TICKS) dead = true;
        __builtin_amdgcn_s_sleep(2);
    }
    __builtin_amdgcn_fence(__ATOMIC_ACQUIRE, "");
    if (dead || __hip_atomic_load(tmo, __ATOMIC_RELAXED, __HIP_MEMORY_SCOPE_AGENT) == call) {
        if (lane == 0) {
            __hip_atomic_store(tmo, call, __ATOMIC_RELAXED, __HIP_MEMORY_SCOPE_AGENT);
            p.out[NR - 1][(size_t)(8 * (p.rank & 1) * 256) * N + (NREG + (p.rank >> 1)) * 256] = __builtin_nanf("");
        }
    }
    STAMP(p, 15);
}

struct Tile { int i, j, need0, step, sh; };

__device__ __forceinline__ void tile_of(int rank, int xcd, int e, Tile& t) {
    if (xcd >= 6 && e >= 31) {
        if (e < 35) {
            t.sh = (xcd - 6) * 4 + e - 31;
            t.i = 8 * (rank & 1) + t.sh; t.j = NREG + (rank >> 1);
            t.need0 = 129 + (rank >> 1); t.step = 20;
            return;
        }
        e -= 4;
    }
    t.sh = -1;
    t.i = 4 * (xcd & 3) + (e & 3);
    t.j = 2 * (e >> 2) + (xcd >> 2);
    if (t.j < 16) { t.need0 = 1 + (2 * t.i > 2 * t.j + 1 ? 2 * t.i : 2 * t.j + 1); t.step = 32; }
    else if (t.j < 32) { t.need0 = 117 + t.j; t.step = 20; }
    else if (t.j < 48) { t.need0 = 177 + t.j; t.step = 16; }
    else { t.need0 = 225 + t.j; t.step = 12; }
}

__device__ __forceinline__ void gemm_role(const Params& p, LAS unsigned char* lds, int xcd, int rk) {
    const unsigned call = (unsigned)__builtin_amdgcn_readfirstlane((int)load_epoch(p)) + 1u;
    const int tid = threadIdx.x, wid = __builtin_amdgcn_readfirstlane(tid >> 6), lane = tid & 63;
    const int wr = wid >> 2, wc = wid & 3;
    const unsigned char* aimg = p.ws[NR - 1] + WS_IMG + (size_t)(call & 1u) * IMG;
    const __amdgpu_buffer_rsrc_t ri = __builtin_amdgcn_make_buffer_rsrc((void*)aimg, 0, (int)IMG, 0x00020000);
    const unsigned* prog = (const unsigned*)(p.ws[NR - 1] + WS_PROG);
    unsigned* tmo = (unsigned*)(p.ws[NR - 1] + WS_TMO);
    LAS volatile unsigned* ctrl = (LAS volatile unsigned*)(lds + LDS_CTRL);
    const unsigned pbase = (call - 1u) * NBLK;
    unsigned voff0 = (unsigned)tid * 16u, voff1 = voff0 + 8192u;
    asm volatile("" : "+v"(voff0), "+v"(voff1));
    const unsigned ldsw = (unsigned)wid * 1024u;
    int aoff = wr * 8192 + lane * 16, boff = 4 * HTB + wc * 4096 + lane * 16;
    asm volatile("" : "+v"(aoff), "+v"(boff));
    const int rank = p.rank, nq = xcd < 6 ? 30 : 31;
    unsigned vprog = pbase;
    const unsigned* progl = prog + (lane & 7);

#define SA(b, h) (((b) * 2 + (h)) * HTB)
#define SB(b, h) ((4 + (b) * 2 + (h)) * HTB)
#define STAGE(bufoff, goff) do { const int _g = (goff); \
        __builtin_amdgcn_raw_ptr_buffer_load_lds(ri, (LAS void*)(lds + (bufoff) + ldsw), 16, voff0, _g, 0, 0); \
        __builtin_amdgcn_raw_ptr_buffer_load_lds(ri, (LAS void*)(lds + (bufoff) + ldsw + 8192), 16, voff1, _g, 0, 0); } while (0)
#define FRAG(off) __builtin_shufflevector(*(const LAS i32x4*)(lds + (off)), *(const LAS i32x4*)(lds + (off) + 1024), 0, 1, 2, 3, 4, 5, 6, 7)
#define LDA(dst, b, h) do { _Pragma("unroll") for (int m = 0; m < 4; ++m) dst[m] = FRAG(SA(b, h) + aoff + m * 2048); } while (0)
#define LDB(dst, b, h) do { _Pragma("unroll") for (int n = 0; n < 2; ++n) dst[n] = FRAG(SA(b, h) + boff + n * 2048); } while (0)
#define MMA(ai, bj, At, Bt, Z) do { __builtin_amdgcn_s_setprio(1); _Pragma("unroll") for (int m = 0; m < 4; ++m) _Pragma("unroll") for (int nn = 0; nn < 2; ++nn) { \
        const int n = SNAKE ? nn ^ (m & 1) : nn; \
        acc[ai][bj][m][n] = __builtin_amdgcn_mfma_scale_f32_16x16x128_f8f6f4(At[m], Bt[n], (Z) ? (f32x4){0.f, 0.f, 0.f, 0.f} : acc[ai][bj][m][n], 0, 0, 0, 0, 0, 0); \
        if (SNAKE) __builtin_amdgcn_sched_barrier(0); } \
        __builtin_amdgcn_s_setprio(0); } while (0)
#define WAIT_V(n) asm volatile("s_waitcnt vmcnt(" #n ")" ::: "memory")
#define WAIT_L(n) asm volatile("s_waitcnt lgkmcnt(" #n ")" ::: "memory")
#define BAR __builtin_amdgcn_s_barrier()
#define SCHED __builtin_amdgcn_sched_barrier(0)
#define KOFF(t) ((((rank + (t)) & 7) * 4 + ((t) >> 3)) * BLK)
#define GATE(need0, step, t) do { if (GATE_ON && wid == 0) { \
        const int _src = (rank + (t)) & 7; \
        const unsigned _need = pbase + (unsigned)((need0) + ((t) >> 3) * (step)); \
        if ((int)((unsigned)__builtin_amdgcn_readlane((int)vprog, _src) - _need) < 0) { \
            const unsigned long long _t0 = __builtin_amdgcn_s_memrealtime(); \
            for (;;) { \
                vprog = __hip_atomic_load(progl, __ATOMIC_RELAXED, __HIP_MEMORY_SCOPE_SYSTEM); \
                if ((int)((unsigned)__builtin_amdgcn_readlane((int)vprog, _src) - _need) >= 0) break; \
                if (__hip_atomic_load(tmo, __ATOMIC_RELAXED, __HIP_MEMORY_SCOPE_AGENT) == call || \
                    __builtin_amdgcn_s_memrealtime() - _t0 > DEADLINE_TICKS) { \
                    __hip_atomic_store(tmo, call, __ATOMIC_RELAXED, __HIP_MEMORY_SCOPE_AGENT); \
                    ctrl[0] = 1u; break; } \
                __builtin_amdgcn_s_sleep(4); \
            } \
            __builtin_amdgcn_s_waitcnt(0xC07F); \
            asm volatile("" ::: "memory"); \
        } } } while (0)
#define POLL_ISSUE() do { if (wid == 0) asm volatile("global_load_dword %0, %1, off sc0 sc1" : "+v"(vprog) : "v"(progl) : "memory"); } while (0)
#define POLL_LANDED() asm volatile("" : "+v"(vprog))

#define ST8(ai, mh, AUX) { float _sc = osc; int _vo = vo; asm volatile("" : "+v"(_sc), "+v"(_vo)); \
        _Pragma("unroll") for (int m = 2 * (mh); m < 2 * (mh) + 2; ++m) _Pragma("unroll") for (int e = 0; e < 4; ++e) { \
            const f32x4 v = {acc[ai][0][m][0][e] * _sc, acc[ai][0][m][1][e] * _sc, acc[ai][1][m][0][e] * _sc, acc[ai][1][m][1][e] * _sc}; \
            __builtin_amdgcn_raw_buffer_store_b128(__builtin_bit_cast(u32x4, v), ro, _vo + ((ai) * 128 + m * 16 + e) * (N * 4), 0, AUX); } }
#define STORES(ai, mh) do { if (osh < 0) ST8(ai, mh, OUT_AUX) else ST8(ai, mh, 16) } while (0)
#define ITER(FIRST, LAST) do { \
        const int t2 = (t + 2) & (NKS - 1), t3 = t2 + 1; \
        const int a1 = cA + KOFF(t + 1); \
        const int a2 = ((LAST) ? nA : cA) + KOFF(t2); \
        const int b2 = ((LAST) ? nB : cB) + KOFF(t2); \
        const int a3 = ((LAST) ? nA : cA) + KOFF(t3); \
        const int b3 = ((LAST) ? nB : cB) + KOFF(t3); \
        GATE((LAST) ? nxt.need0 : cur.need0, (LAST) ? nxt.step : cur.step, t2); \
        POLL_ISSUE(); \
        LDB(B0, 0, 0); SCHED; LDA(At, 0, 0); STAGE(SA(1, 1), a1 + HTB); \
        if (FIRST) STORES(1, 0); \
        WAIT_L(8); BAR; WAIT_L(0); MMA(0, 0, At, B0, FIRST); BAR; SCHED; \
        LDB(B1, 0, 1); STAGE(SB(0, 0), b2); \
        if (FIRST) STORES(1, 1); \
        BAR; WAIT_L(0); MMA(0, 1, At, B1, FIRST); BAR; \
        LDA(At, 0, 1); STAGE(SA(0, 0), a2); \
        BAR; WAIT_L(0); MMA(1, 0, At, B0, FIRST); BAR; SCHED; \
        STAGE(SB(0, 1), b2 + HTB); \
        if (FIRST) WAIT_V(22); else WAIT_V(6); \
        POLL_LANDED(); BAR; MMA(1, 1, At, B1, FIRST); BAR; \
        GATE((LAST) ? nxt.need0 : cur.need0, (LAST) ? nxt.step : cur.step, t3); \
        POLL_ISSUE(); \
        unsigned dead = 0u; \
        if (LAST) dead = ctrl[0]; \
        LDB(B0, 1, 0); SCHED; LDA(At, 1, 0); STAGE(SA(0, 1), a2 + HTB); \
        WAIT_L(8); BAR; WAIT_L(0); MMA(0, 0, At, B0, 0); BAR; SCHED; \
        LDB(B1, 1, 1); STAGE(SB(1, 0), b3); \
        BAR; WAIT_L(0); MMA(0, 1, At, B1, 0); BAR; \
        LDA(At, 1, 1); STAGE(SA(1, 0), a3); \
        if (LAST) { \
            ro = __builtin_amdgcn_make_buffer_rsrc(p.out[NR - 1] + (size_t)(cur.i * 256 + wr * 64) * N + cur.j * 256 + wc * 64, 0, 192 * N * 4, 0x00020000); \
            osh = cur.sh; osc = dead ? __builtin_nanf("") : scale; \
            STORES(0, 0); \
        } \
        BAR; WAIT_L(0); MMA(1, 0, At, B0, 0); BAR; SCHED; \
        STAGE(SB(1, 1), b3 + HTB); \
        if (LAST) WAIT_V(14); else WAIT_V(6); \
        POLL_LANDED(); \
        if (LAST) STORES(0, 1); \
        if (FIRST) { if (osh >= 0 && lane == 0) __hip_atomic_fetch_add(tcnt + osh * 32, 1u, __ATOMIC_RELAXED, __HIP_MEMORY_SCOPE_AGENT); } \
        BAR; MMA(1, 1, At, B1, 0); BAR; \
    } while (0)

    Tile cur, nxt;
    tile_of(rank, xcd, rk, cur);
    f32x4 acc[2][2][4][2];
#pragma unroll
    for (int b = 0; b < 2; ++b)
#pragma unroll
        for (int m = 0; m < 4; ++m)
#pragma unroll
            for (int n = 0; n < 2; ++n) acc[1][b][m][n] = (f32x4){0.f, 0.f, 0.f, 0.f};
    i32x8 At[4], B0[2], B1[2];
    int cA = cur.i * NKS * BLK, cB = (int)AIMG + cur.j * NKS * BLK;
    const float scale = p.sx[0] * p.sw[0];
    unsigned* tcnt = (unsigned*)(p.ws[NR - 1] + WS_TCNT);
    __amdgpu_buffer_rsrc_t ro = __builtin_amdgcn_make_buffer_rsrc(p.out[NR - 1], 0, 0, 0x00020000);
    int osh = -1;
    float osc = scale;
    const int vo = (lane >> 4) * (4 * N * 4) + (lane & 15) * 16;

    {
        for (int t = ((int)blockIdx.x - NCOMM) * 8 + wid; t < NTASK_IN; t += (NWG - NCOMM) * 8) {
            unsigned raw[16], off[4];
            Task k; u32x4 o[4];
            task_decode(rank, t, lane, k);
            task_load(p, k, lane, raw);
            task_tile(k, raw, o, off);
            if (k.only < 0 || k.only == (rank >> 1)) {
#pragma unroll
                for (int q = 0; q < 4; ++q) __builtin_amdgcn_raw_buffer_store_b128(o[q], ri, off[q], 0, 16);
            }
        }
        WAIT_V(0);
        BAR;
        if (tid == 0) {
            const unsigned n = __hip_atomic_fetch_add((unsigned*)(p.ws[NR - 1] + WS_LCNT), 1u, __ATOMIC_RELAXED, __HIP_MEMORY_SCOPE_AGENT) + 1u;
            if (n == call * (unsigned)(NWG - NCOMM))
                __hip_atomic_store((unsigned*)(p.ws[NR - 1] + WS_PROG) + rank, pbase + NBLK, __ATOMIC_RELAXED, __HIP_MEMORY_SCOPE_SYSTEM);
            ctrl[0] = 0u;
            __builtin_amdgcn_fence(__ATOMIC_ACQUIRE, "");
            asm volatile("s_waitcnt vmcnt(0) lgkmcnt(0)" ::: "memory");
        }
    }
    GATE(cur.need0, cur.step, 0); GATE(cur.need0, cur.step, 1);
    BAR;
    STAGE(SB(0, 0), cB + KOFF(0)); STAGE(SA(0, 0), cA + KOFF(0)); STAGE(SB(0, 1), cB + KOFF(0) + HTB); STAGE(SA(0, 1), cA + KOFF(0) + HTB);
    if (wr == 1) BAR;
    WAIT_V(4); BAR;
    STAGE(SB(1, 0), cB + KOFF(1)); STAGE(SA(1, 0), cA + KOFF(1)); STAGE(SB(1, 1), cB + KOFF(1) + HTB);
    WAIT_V(6); BAR;
    if (wid == 0) STAMP(p, 1);
#pragma unroll 1
    for (int un = 0; un < 4; ++un) {
        tile_of(rank, xcd, rk + (un < 3 ? un + 1 : un) * nq, nxt);
        const int nA = nxt.i * NKS * BLK, nB = (int)AIMG + nxt.j * NKS * BLK;
        int t = 0;
        ITER(1, 0);
#pragma unroll 1
        for (t = 2; t < NKS - 2; t += 2) ITER(0, 0);
        ITER(0, 1);
        if (wid == 0) STAMP(p, 2 + 2 * un);
        cur = nxt; cA = nA; cB = nB;
    }
    STORES(1, 0);
    STORES(1, 1);
    if (wid == 0) STAMP(p, 9);
    WAIT_V(0);
    if (wr == 0) BAR;
#undef ST8
#undef STORES
#undef ITER
#undef SA
#undef SB
#undef STAGE
#undef FRAG
#undef LDA
#undef LDB
#undef MMA
#undef WAIT_V
#undef WAIT_L
#undef BAR
#undef SCHED
#undef KOFF
#undef GATE
#undef POLL_ISSUE
#undef POLL_LANDED
}

__global__ void __launch_bounds__(512, 2) fused_gemm_allgather(const Params p) {
    extern __shared__ __attribute__((aligned(16))) unsigned char shm[];
    LAS unsigned char* lds = (LAS unsigned char*)shm;
    const int b = blockIdx.x;
    if (threadIdx.x == 0) STAMP(p, 0);
    if (b < NCOMM) {
        const int wid = __builtin_amdgcn_readfirstlane((int)(threadIdx.x >> 6)), lane = threadIdx.x & 63;
        bool run = true;
        if (run) {
            const int cw = b == 0 ? (wid >= 1 && wid <= TW0 ? wid - 1 : -1) : (wid < TW ? TW0 + (b - 1) * TW + wid : -1);
            if (cw >= 0) comm_tasks(p, lds + wid * 4096, cw, lane);
            else if (b == 0 && wid == 0) comm_publish(p, lane);
        }
    } else {
        const int xcd = b & 7;
        gemm_role(p, lds, xcd, (b >> 3) - (xcd < 6 ? 2 : 1));
    }
    __builtin_amdgcn_s_barrier();
    if (threadIdx.x == 0) {
        const unsigned ticket = __hip_atomic_fetch_add((unsigned*)(p.ws[NR - 1] + WS_TICKET), 1u, __ATOMIC_RELAXED, __HIP_MEMORY_SCOPE_AGENT) + 1u;
        if (ticket % NWG == 0u) __hip_atomic_store((unsigned*)(p.ws[NR - 1] + WS_EPOCH), ticket / NWG, __ATOMIC_RELAXED, __HIP_MEMORY_SCOPE_AGENT);
    }
}

}

extern "C" void kernel_launch(int rank, int nranks, ncclComm_t comm,
                              hipStream_t stream,
                              void* const* d_in, const int* in_sizes, int n_in,
                              void* d_out, int out_size,
                              const int* in_dtypes, int out_dtype,
                              void* const* d_out_all,
                              void* const* d_ws_all, size_t ws_bytes) {
    (void)comm; (void)in_sizes; (void)in_dtypes; (void)out_dtype; (void)d_out;
    if (nranks != NR || n_in != 4 || (unsigned)rank >= (unsigned)NR || ws_bytes < WS_NEED || out_size != M * N) {
        fprintf(stderr, "kernel_launch: unexpected problem shape\n");
        return;
    }
    Params p;
    p.x = (const unsigned char*)d_in[0];
    p.w = (const unsigned char*)d_in[1];
    p.sx = (const float*)d_in[2];
    p.sw = (const float*)d_in[3];
    for (int i = 0; i < NR; ++i) {
        p.ws[i] = (unsigned char*)d_ws_all[(rank + 1 + i) % NR];
        p.out[i] = (float*)d_out_all[(rank + 1 + i) % NR];
    }
    p.rank = rank;
    HIP_CHECK(hipFuncSetAttribute((const void*)fused_gemm_allgather, hipFuncAttributeMaxDynamicSharedMemorySize, LDS_BYTES));
    hipLaunchKernelGGL(fused_gemm_allgather, dim3(NWG), dim3(512), LDS_BYTES, stream, p);
    HIP_CHECK(hipGetLastError());
}
```

```cpp
#include <hip/hip_runtime.h>
#include <rccl/rccl.h>
#include <cstdio>
#include <cstdint>

#define HIP_CHECK(expr) do { hipError_t _e = (expr); if (_e != hipSuccess) { \
    fprintf(stderr, "HIP error %s at %s:%d\n", hipGetErrorString(_e), __FILE__, __LINE__); \
    return; } } while (0)

namespace {

#define LAS __attribute__((address_space(3)))
typedef int i32x4 __attribute__((ext_vector_type(4)));
typedef int i32x8 __attribute__((ext_vector_type(8)));
typedef unsigned u32x4 __attribute__((ext_vector_type(4)));
typedef float f32x4 __attribute__((ext_vector_type(4)));

constexpr int M = 4096, N = 16384, NR = 8, KS = 512;
constexpr int NKS = 32;
constexpr int BLK = 32768;
constexpr int HTB = 16384;
constexpr int NSH = 4;
constexpr int NMT = 16, NNT = 64, NREG = NNT - NSH;
constexpr int TPR = 2 * NSH;
constexpr size_t AIMG = (size_t)NMT * NKS * BLK, BIMG = (size_t)NNT * NKS * BLK, IMG = AIMG + BIMG;

constexpr int G1 = 16 + NSH, G3 = NREG - 48;
constexpr int P2 = 128 + 4 * G1, P3 = P2 + 64;
constexpr int NBLK_IN = 320, NBLK = NBLK_IN + 8 * TPR;
constexpr int NTASK_IN = NBLK_IN * 8, NTASK = NBLK * 8;
constexpr int TW = 2;
constexpr int AUX_NT = 2, AUX_SC1 = 16, AUX_SYS = 17;
constexpr int NWG = 256, NCOMM = 14;
constexpr int NCW = NCOMM * TW;

constexpr size_t WS_TICKET = 0, WS_EPOCH = 128, WS_TMO = 256, WS_PROG = 512, WS_CLAIM = 768, WS_LCNT = 896, WS_TCNT = 1024, WS_DONE = 4096, WS_IMG = 1u << 20;
constexpr size_t WS_NEED = WS_IMG + 2 * IMG;
constexpr unsigned long long DEADLINE_TICKS = 150000000ull;

constexpr int LDS_CTRL = 8 * HTB;
constexpr int LDS_BYTES = LDS_CTRL + 64;

struct Params {
    const unsigned char* x;
    const unsigned char* w;
    const float* sx;
    const float* sw;
    unsigned char* ws[NR];
    float* out[NR];
    int rank;
};

__device__ __forceinline__ bool owns_column(int rank, int c) { return TPR * rank < 16 * c + 16 && TPR * rank + TPR > 16 * c; }

__device__ __forceinline__ void transpose4(unsigned d0, unsigned d1, unsigned d2, unsigned d3,
                                           unsigned& o0, unsigned& o1, unsigned& o2, unsigned& o3) {
    const unsigned t0 = __builtin_amdgcn_perm(d1, d0, 0x05010400u), t1 = __builtin_amdgcn_perm(d1, d0, 0x07030602u);
    const unsigned t2 = __builtin_amdgcn_perm(d3, d2, 0x05010400u), t3 = __builtin_amdgcn_perm(d3, d2, 0x07030602u);
    o0 = __builtin_amdgcn_perm(t2, t0, 0x05040100u); o1 = __builtin_amdgcn_perm(t2, t0, 0x07060302u);
    o2 = __builtin_amdgcn_perm(t3, t1, 0x05040100u); o3 = __builtin_amdgcn_perm(t3, t1, 0x07060302u);
}

__device__ __forceinline__ unsigned load_epoch(const Params& p) {
    return __hip_atomic_load((const unsigned*)(p.ws[NR - 1] + WS_EPOCH), __ATOMIC_RELAXED, __HIP_MEMORY_SCOPE_AGENT);
}

struct Task { int kind, only, tile, kk, sub; unsigned base; };

__device__ __forceinline__ void task_decode(int rank, int t, int lane, Task& k) {
    const int bp = t >> 3;
    k.sub = t & 7; k.only = -1; k.kind = 1;
    if (bp < 128) { k.kk = bp >> 5; k.kind = bp & 1; k.tile = (bp & 31) >> 1; }
    else if (bp < P2) {
        k.kk = (bp - 128) / G1; const int q = bp - 128 - k.kk * G1;
        if (q < NSH) { k.tile = NREG + q; k.only = q; } else k.tile = 16 - NSH + q;
    } else if (bp < P3) { const int q = bp - P2; k.kk = q >> 4; k.tile = 32 + (q & 15); }
    else if (bp < NBLK_IN) { const int q = bp - P3; k.kk = q / G3; k.tile = 48 + (q - k.kk * G3); }
    else { k.kind = 2; k.kk = 0; k.tile = (bp - NBLK_IN) >> 3; }
    const int z = TPR * rank + k.tile;
    const unsigned blk = (unsigned)(k.tile * NKS + rank * 4 + k.kk) * BLK + lane * 16;
    k.base = k.kind == 0 ? blk + k.sub * 4096
           : k.kind == 1 ? (unsigned)AIMG + blk + (k.sub >> 1) * 4096 + (k.sub & 1) * 1024
                         : (unsigned)((z & 15) * 256 + ((t - NTASK_IN) & 63) * 4) * (unsigned)(N * 4) + (NREG + (z >> 4)) * 1024 + lane * 16;
}

__device__ __forceinline__ void task_load(const Params& p, const Task& k, int lane, unsigned (&raw)[16]) {
    const int r = lane & 15, g = lane >> 4;
    if (k.kind == 0) {
        const unsigned char* src = p.x + (size_t)(256 * k.tile + 32 * k.sub + r) * KS + 128 * k.kk + 16 * g;
#pragma unroll
        for (int q = 0; q < 4; ++q) {
            const u32x4 v = *(const u32x4*)(src + (q >> 1) * 16 * KS + (q & 1) * 64);
            raw[4 * q] = v[0]; raw[4 * q + 1] = v[1]; raw[4 * q + 2] = v[2]; raw[4 * q + 3] = v[3];
        }
    } else {
        const unsigned char* src = p.w + (size_t)(128 * k.kk + 64 * (k.sub & 1) + 16 * g) * N + 256 * k.tile + 64 * (k.sub >> 1) + 4 * r;
#pragma unroll
        for (int b = 0; b < 16; ++b) raw[b] = *(const unsigned*)(src + (size_t)b * N);
    }
}

__device__ __forceinline__ void task_tile(const Task& k, const unsigned (&raw)[16], u32x4 (&o)[4], unsigned (&off)[4]) {
    if (k.kind != 1) {
        const unsigned step = k.kind == 0 ? 1024u : (unsigned)(N * 4);
#pragma unroll
        for (int q = 0; q < 4; ++q) { o[q] = (u32x4){raw[4 * q], raw[4 * q + 1], raw[4 * q + 2], raw[4 * q + 3]}; off[q] = k.base + q * step; }
    } else {
#pragma unroll
        for (int a = 0; a < 4; ++a) {
            unsigned o0, o1, o2, o3;
            transpose4(raw[4 * a], raw[4 * a + 1], raw[4 * a + 2], raw[4 * a + 3], o0, o1, o2, o3);
            o[0][a] = o0; o[1][a] = o1; o[2][a] = o2; o[3][a] = o3;
        }
#pragma unroll
        for (int q = 0; q < 4; ++q) off[q] = k.base + (q >> 1) * HTB + (q & 1) * 2048;
    }
}

__device__ __forceinline__ void comm_tasks(const Params& p, LAS unsigned char* stage, int cw, int lane) {
    unsigned epoch = load_epoch(p), call = 0u;
    unsigned* done = (unsigned*)(p.ws[NR - 1] + WS_DONE);
    unsigned* claim = (unsigned*)(p.ws[NR - 1] + WS_CLAIM);
    const unsigned* tcnt = (const unsigned*)(p.ws[NR - 1] + WS_TCNT);
    unsigned* tmo = (unsigned*)(p.ws[NR - 1] + WS_TMO);
    const unsigned sbase = (unsigned)(size_t)stage;
    const int r = lane & 15, g = lane >> 4;
    int ready = -1;
#define FETCH_TASK(k) do { \
        const unsigned char* _s; size_t _d1, _d2; \
        if (k.kind == 0) { _s = p.x + (size_t)(256 * k.tile + 32 * k.sub + r) * KS + 128 * k.kk + 16 * g; _d1 = 64; _d2 = 16 * KS; } \
        else if (k.kind == 1) { _s = p.w + (size_t)(128 * k.kk + 64 * (k.sub & 1) + (lane >> 2)) * N + 256 * k.tile + 64 * (k.sub >> 1) + 16 * (lane & 3); \
            _d1 = (size_t)16 * N; _d2 = (size_t)32 * N; } \
        else { \
            if (k.tile > ready) { \
                const unsigned long long _t0 = __builtin_amdgcn_s_memrealtime(); \
                while ((int)(__hip_atomic_load(tcnt + k.tile * 32, __ATOMIC_RELAXED, __HIP_MEMORY_SCOPE_AGENT) - 8u * call) < 0) { \
                    if (__hip_atomic_load(tmo, __ATOMIC_RELAXED, __HIP_MEMORY_SCOPE_AGENT) == call || \
                        __builtin_amdgcn_s_memrealtime() - _t0 > DEADLINE_TICKS) { \
                        __hip_atomic_store(tmo, call, __ATOMIC_RELAXED, __HIP_MEMORY_SCOPE_AGENT); \
                        break; } \
                    __builtin_amdgcn_s_sleep(8); \
                } \
                __builtin_amdgcn_fence(__ATOMIC_ACQUIRE, "agent"); \
                __builtin_amdgcn_s_waitcnt(0x0070); \
                ready = k.tile; \
            } \
            _s = (const unsigned char*)p.out[NR - 1] + k.base; _d1 = (size_t)N * 4; _d2 = (size_t)N * 8; \
        } \
        unsigned _m0; \
        asm volatile("s_mov_b32 %0, m0\n\ts_mov_b32 m0, %5\n\ts_nop 0\n\tglobal_load_lds_dwordx4 %1, off\n\t" \
                     "s_add_u32 m0, m0, 0x400\n\ts_nop 0\n\tglobal_load_lds_dwordx4 %2, off\n\t" \
                     "s_add_u32 m0, m0, 0x400\n\ts_nop 0\n\tglobal_load_lds_dwordx4 %3, off\n\t" \
                     "s_add_u32 m0, m0, 0x400\n\ts_nop 0\n\tglobal_load_lds_dwordx4 %4, off\n\ts_mov_b32 m0, %0" \
                     : "=&s"(_m0) : "v"(_s), "v"(_s + _d1), "v"(_s + _d2), "v"(_s + _d1 + _d2), "s"(sbase) : "memory", "scc"); } while (0)
    int prev = -1, cur = cw, nxt = cw + NCW;
    Task k;
    task_decode(p.rank, cur, lane, k);
    FETCH_TASK(k);
    asm volatile("s_waitcnt vmcnt(0)" : "+v"(epoch) :: "memory");
    call = (unsigned)__builtin_amdgcn_readfirstlane((int)epoch) + 1u;
    const unsigned cbase = (call - 1u) * (unsigned)NTASK;
    const unsigned par = (call & 1u) * (unsigned)IMG;
    for (;;) {
        unsigned raw[16], off[4];
        u32x4 o[4];
        if (k.kind == 1) {
#pragma unroll
            for (int b = 0; b < 16; ++b) raw[b] = *(const LAS unsigned*)(stage + g * 1024 + b * 64 + r * 4);
        } else {
#pragma unroll
            for (int q = 0; q < 4; ++q) {
                const u32x4 v = *(const LAS u32x4*)(stage + q * 1024 + lane * 16);
                raw[4 * q] = v[0]; raw[4 * q + 1] = v[1]; raw[4 * q + 2] = v[2]; raw[4 * q + 3] = v[3];
            }
        }
        task_tile(k, raw, o, off);
        asm volatile("s_waitcnt lgkmcnt(0)" : "+v"(o[0]), "+v"(o[1]), "+v"(o[2]), "+v"(o[3]) :: "memory");
        const bool is_out = k.kind == 2;
        const unsigned add = is_out ? 0u : par;
        __amdgpu_buffer_rsrc_t rs[NR - 1];
#pragma unroll
        for (int d = 0; d < NR - 1; ++d) {
            const bool want = k.only < 0 || owns_column((p.rank + 1 + d) & 7, k.only);
            rs[d] = __builtin_amdgcn_make_buffer_rsrc(is_out ? (unsigned char*)p.out[d] : p.ws[d] + WS_IMG, 0,
                                                      want ? (is_out ? M * N * 4 : (int)(2 * IMG)) : 0, 0x00020000);
        }
        const bool more = (unsigned)nxt < (unsigned)NTASK;
        task_decode(p.rank, more ? nxt : cur, lane, k);
        FETCH_TASK(k);
        unsigned tk = 1u; unsigned long long ex;
        asm volatile("s_mov_b64 %1, exec\n\ts_mov_b64 exec, 1\n\tglobal_atomic_add %0, %2, %0, off sc0\n\ts_mov_b64 exec, %1"
                     : "+v"(tk), "=&s"(ex) : "v"(claim) : "memory");
#pragma unroll
        for (int q = 0; q < 4; ++q)
#pragma unroll
            for (int d = 0; d < NR - 1; ++d) __builtin_amdgcn_raw_buffer_store_b128(o[q], rs[d], off[q] + add, 0, AUX_SYS);
        asm volatile("s_waitcnt vmcnt(28)" : "+v"(tk) :: "memory");
        const int got = 2 * NCW + (int)((unsigned)__builtin_amdgcn_readfirstlane((int)tk) - cbase);
        if (prev >= 0 && lane == 0) __hip_atomic_store(done + prev, call, __ATOMIC_RELAXED, __HIP_MEMORY_SCOPE_AGENT);
        prev = cur;
        if (!more) break;
        cur = nxt;
        nxt = got;
    }
    asm volatile("s_waitcnt vmcnt(0)" ::: "memory");
    if (lane == 0) __hip_atomic_store(done + prev, call, __ATOMIC_RELAXED, __HIP_MEMORY_SCOPE_AGENT);
#undef FETCH_TASK
}

__device__ __forceinline__ void comm_publish(const Params& p, int lane) {
    const unsigned call = (unsigned)__builtin_amdgcn_readfirstlane((int)load_epoch(p)) + 1u;
    const unsigned* done = (const unsigned*)(p.ws[NR - 1] + WS_DONE);
    unsigned* tmo = (unsigned*)(p.ws[NR - 1] + WS_TMO);
    unsigned* dst = (unsigned*)(p.ws[0] + WS_PROG);
#pragma unroll
    for (int d = 1; d < NR - 1; ++d) if (lane == d) dst = (unsigned*)(p.ws[d] + WS_PROG);
    dst += p.rank;
    const unsigned base = (call - 1u) * NBLK;
    const unsigned long long t0 = __builtin_amdgcn_s_memrealtime();
    int nt = 0, pub = 0;
    bool dead = false;
    while (nt < NTASK) {
        const int idx = nt + lane;
        unsigned v = 0;
        if (idx < NTASK) v = __hip_atomic_load(done + idx, __ATOMIC_RELAXED, __HIP_MEMORY_SCOPE_AGENT);
        const unsigned long long miss = ~__ballot(v == call);
        const int n = miss ? __builtin_ctzll(miss) : 64;
        nt += n;
        if ((nt >> 3) > pub) {
            pub = nt >> 3;
            if (lane < NR - 1) __hip_atomic_store(dst, base + (unsigned)pub, __ATOMIC_RELAXED, __HIP_MEMORY_SCOPE_SYSTEM);
        }
        if (n == 0) {
            if (__builtin_amdgcn_s_memrealtime() - t0 > DEADLINE_TICKS) { dead = true; break; }
            __builtin_amdgcn_s_sleep(2);
        }
    }
    const unsigned* progl = (const unsigned*)(p.ws[NR - 1] + WS_PROG) + (lane & 7);
    while (!dead) {
        const unsigned v = __hip_atomic_load(progl, __ATOMIC_RELAXED, __HIP_MEMORY_SCOPE_SYSTEM);
        if (__all((int)(v - (base + NBLK)) >= 0)) break;
        if (__builtin_amdgcn_s_memrealtime() - t0 > DEADLINE_TICKS) dead = true;
        __builtin_amdgcn_s_sleep(2);
    }
    __builtin_amdgcn_fence(__ATOMIC_ACQUIRE, "");
    if (dead || __hip_atomic_load(tmo, __ATOMIC_RELAXED, __HIP_MEMORY_SCOPE_AGENT) == call) {
        if (lane == 0) {
            __hip_atomic_store(tmo, call, __ATOMIC_RELAXED, __HIP_MEMORY_SCOPE_AGENT);
            p.out[NR - 1][0] = __builtin_nanf("");
        }
    }
}

struct Tile { int i, j, need0, step, sh; };

__device__ __forceinline__ void tile_of(int rank, int xcd, int e, Tile& t) {
    const int first = 31, ns = xcd >= 6 ? 4 : 0, u0 = (xcd - 6) * 4;
    if (e >= first) {
        if (e < first + ns) {
            t.sh = u0 + e - first;
            const int z = TPR * rank + t.sh;
            t.i = z & 15; t.j = NREG + (z >> 4);
            t.need0 = 129 + (z >> 4); t.step = G1;
            return;
        }
        e -= ns;
    }
    t.sh = -1;
    t.i = 4 * (xcd & 3) + (e & 3);
    t.j = 2 * (e >> 2) + (xcd >> 2);
    if (t.j < 16) { t.need0 = 1 + (2 * t.i > 2 * t.j + 1 ? 2 * t.i : 2 * t.j + 1); t.step = 32; }
    else if (t.j < 32) { t.need0 = 113 + NSH + t.j; t.step = G1; }
    else if (t.j < 48) { t.need0 = P2 - 31 + t.j; t.step = 16; }
    else { t.need0 = P3 - 47 + t.j; t.step = G3; }
}

__device__ __forceinline__ void gemm_role(const Params& p, LAS unsigned char* lds, int xcd, int rk) {
    const unsigned call = (unsigned)__builtin_amdgcn_readfirstlane((int)load_epoch(p)) + 1u;
    const int tid = threadIdx.x, wid = __builtin_amdgcn_readfirstlane(tid >> 6), lane = tid & 63;
    const int wr = wid >> 2, wc = wid & 3;
    const unsigned char* aimg = p.ws[NR - 1] + WS_IMG + (size_t)(call & 1u) * IMG;
    const __amdgpu_buffer_rsrc_t ri = __builtin_amdgcn_make_buffer_rsrc((void*)aimg, 0, (int)IMG, 0x00020000);
    const unsigned* prog = (const unsigned*)(p.ws[NR - 1] + WS_PROG);
    unsigned* tmo = (unsigned*)(p.ws[NR - 1] + WS_TMO);
    LAS volatile unsigned* ctrl = (LAS volatile unsigned*)(lds + LDS_CTRL);
    const unsigned pbase = (call - 1u) * NBLK;
    unsigned voff0 = (unsigned)tid * 16u, voff1 = voff0 + 8192u;
    asm volatile("" : "+v"(voff0), "+v"(voff1));
    const unsigned ldsw = (unsigned)wid * 1024u;
    int aoff = wr * 8192 + lane * 16, boff = 4 * HTB + wc * 4096 + lane * 16;
    asm volatile("" : "+v"(aoff), "+v"(boff));
    const int rank = p.rank;
    const int nq = xcd < 6 ? 30 : 31;
    unsigned vprog = pbase;
    const unsigned* progl = prog + (lane & 7);

#define SA(b, h) (((b) * 2 + (h)) * HTB)
#define SB(b, h) ((4 + (b) * 2 + (h)) * HTB)
#define STAGE(bufoff, goff) do { const int _g = (goff); \
        __builtin_amdgcn_raw_ptr_buffer_load_lds(ri, (LAS void*)(lds + (bufoff) + ldsw), 16, voff0, _g, 0, AUX_SC1); \
        __builtin_amdgcn_raw_ptr_buffer_load_lds(ri, (LAS void*)(lds + (bufoff) + ldsw + 8192), 16, voff1, _g, 0, AUX_SC1); } while (0)
#define FRAG(off) __builtin_shufflevector(*(const LAS i32x4*)(lds + (off)), *(const LAS i32x4*)(lds + (off) + 1024), 0, 1, 2, 3, 4, 5, 6, 7)
#define LDA(dst, b, h) do { _Pragma("unroll") for (int m = 0; m < 4; ++m) dst[m] = FRAG(SA(b, h) + aoff + m * 2048); } while (0)
#define LDB(dst, b, h) do { _Pragma("unroll") for (int n = 0; n < 2; ++n) dst[n] = FRAG(SA(b, h) + boff + n * 2048); } while (0)
#define MMA(ai, bj, At, Bt, Z) do { __builtin_amdgcn_s_setprio(1); _Pragma("unroll") for (int m = 0; m < 4; ++m) _Pragma("unroll") for (int nn = 0; nn < 2; ++nn) { \
        const int n = nn ^ (m & 1); \
        acc[ai][bj][m][n] = __builtin_amdgcn_mfma_scale_f32_16x16x128_f8f6f4(At[m], Bt[n], (Z) ? (f32x4){0.f, 0.f, 0.f, 0.f} : acc[ai][bj][m][n], 0, 0, 0, 0, 0, 0); \
        __builtin_amdgcn_sched_barrier(0); } \
        __builtin_amdgcn_s_setprio(0); } while (0)
#define WAIT_V(n) asm volatile("s_waitcnt vmcnt(" #n ")" ::: "memory")
#define WAIT_L(n) asm volatile("s_waitcnt lgkmcnt(" #n ")" ::: "memory")
#define BAR __builtin_amdgcn_s_barrier()
#define SCHED __builtin_amdgcn_sched_barrier(0)
#define KOFF(t) ((((rank + (t)) & 7) * 4 + ((t) >> 3)) * BLK)
#define GATE(need0, step, t) do { if (wid == 0) { \
        const int _src = (rank + (t)) & 7; \
        const unsigned _need = pbase + (unsigned)((need0) + ((t) >> 3) * (step)); \
        if ((int)((unsigned)__builtin_amdgcn_readlane((int)vprog, _src) - _need) < 0) { \
            const unsigned long long _t0 = __builtin_amdgcn_s_memrealtime(); \
            for (;;) { \
                vprog = __hip_atomic_load(progl, __ATOMIC_RELAXED, __HIP_MEMORY_SCOPE_SYSTEM); \
                if ((int)((unsigned)__builtin_amdgcn_readlane((int)vprog, _src) - _need) >= 0) break; \
                if (__hip_atomic_load(tmo, __ATOMIC_RELAXED, __HIP_MEMORY_SCOPE_AGENT) == call || \
                    __builtin_amdgcn_s_memrealtime() - _t0 > DEADLINE_TICKS) { \
                    __hip_atomic_store(tmo, call, __ATOMIC_RELAXED, __HIP_MEMORY_SCOPE_AGENT); \
                    ctrl[0] = 1u; break; } \
                __builtin_amdgcn_s_sleep(4); \
            } \
            __builtin_amdgcn_s_waitcnt(0xC07F); \
            asm volatile("" ::: "memory"); \
        } } } while (0)
#define POLL_ISSUE() do { if (wid == 0) asm volatile("global_load_dword %0, %1, off sc0 sc1" : "+v"(vprog) : "v"(progl) : "memory"); } while (0)
#define POLL_LANDED() asm volatile("" : "+v"(vprog))

#define ST8(ai, mh, AUX) { float _sc = osc; int _vo = vo; asm volatile("" : "+v"(_sc), "+v"(_vo)); \
        _Pragma("unroll") for (int m = 2 * (mh); m < 2 * (mh) + 2; ++m) _Pragma("unroll") for (int e = 0; e < 4; ++e) { \
            const f32x4 v = {acc[ai][0][m][0][e] * _sc, acc[ai][0][m][1][e] * _sc, acc[ai][1][m][0][e] * _sc, acc[ai][1][m][1][e] * _sc}; \
            __builtin_amdgcn_raw_buffer_store_b128(__builtin_bit_cast(u32x4, v), ro, _vo + ((ai) * 128 + m * 16 + e) * (N * 4), 0, AUX); } }
#define STORES(ai, mh) do { if (osh < 0) ST8(ai, mh, AUX_NT) else ST8(ai, mh, AUX_SC1) } while (0)
#define ITER(FIRST, LAST) do { \
        const int t2 = (t + 2) & (NKS - 1), t3 = t2 + 1; \
        const int a1 = cA + KOFF(t + 1); \
        const int a2 = ((LAST) ? nA : cA) + KOFF(t2); \
        const int b2 = ((LAST) ? nB : cB) + KOFF(t2); \
        const int a3 = ((LAST) ? nA : cA) + KOFF(t3); \
        const int b3 = ((LAST) ? nB : cB) + KOFF(t3); \
        GATE((LAST) ? nxt.need0 : cur.need0, (LAST) ? nxt.step : cur.step, t2); \
        POLL_ISSUE(); \
        LDB(B0, 0, 0); SCHED; LDA(At, 0, 0); STAGE(SA(1, 1), a1 + HTB); \
        if (FIRST) STORES(1, 0); \
        WAIT_L(8); BAR; WAIT_L(0); MMA(0, 0, At, B0, FIRST); BAR; SCHED; \
        LDB(B1, 0, 1); STAGE(SB(0, 0), b2); \
        if (FIRST) STORES(1, 1); \
        BAR; WAIT_L(0); MMA(0, 1, At, B1, FIRST); BAR; \
        LDA(At, 0, 1); STAGE(SA(0, 0), a2); \
        BAR; WAIT_L(0); MMA(1, 0, At, B0, FIRST); BAR; SCHED; \
        STAGE(SB(0, 1), b2 + HTB); \
        if (FIRST) WAIT_V(22); else WAIT_V(6); \
        POLL_LANDED(); BAR; MMA(1, 1, At, B1, FIRST); BAR; \
        GATE((LAST) ? nxt.need0 : cur.need0, (LAST) ? nxt.step : cur.step, t3); \
        POLL_ISSUE(); \
        unsigned dead = 0u; \
        if (LAST) dead = ctrl[0]; \
        LDB(B0, 1, 0); SCHED; LDA(At, 1, 0); STAGE(SA(0, 1), a2 + HTB); \
        WAIT_L(8); BAR; WAIT_L(0); MMA(0, 0, At, B0, 0); BAR; SCHED; \
        LDB(B1, 1, 1); STAGE(SB(1, 0), b3); \
        BAR; WAIT_L(0); MMA(0, 1, At, B1, 0); BAR; \
        LDA(At, 1, 1); STAGE(SA(1, 0), a3); \
        if (LAST) { \
            ro = __builtin_amdgcn_make_buffer_rsrc(p.out[NR - 1] + (size_t)(cur.i * 256 + wr * 64) * N + cur.j * 256 + wc * 64, 0, 192 * N * 4, 0x00020000); \
            osh = cur.sh; osc = dead ? __builtin_nanf("") : scale; \
            STORES(0, 0); \
        } \
        BAR; WAIT_L(0); MMA(1, 0, At, B0, 0); BAR; SCHED; \
        STAGE(SB(1, 1), b3 + HTB); \
        if (LAST) WAIT_V(14); else WAIT_V(6); \
        POLL_LANDED(); \
        if (LAST) STORES(0, 1); \
        if (FIRST) { if (osh >= 0 && lane == 0) __hip_atomic_fetch_add(tcnt + osh * 32, 1u, __ATOMIC_RELAXED, __HIP_MEMORY_SCOPE_AGENT); } \
        BAR; MMA(1, 1, At, B1, 0); BAR; \
    } while (0)

    Tile cur, nxt;
    tile_of(rank, xcd, rk, cur);
    f32x4 acc[2][2][4][2];
#pragma unroll
    for (int b = 0; b < 2; ++b)
#pragma unroll
        for (int m = 0; m < 4; ++m)
#pragma unroll
            for (int n = 0; n < 2; ++n) acc[1][b][m][n] = (f32x4){0.f, 0.f, 0.f, 0.f};
    i32x8 At[4], B0[2], B1[2];
    int cA = cur.i * NKS * BLK, cB = (int)AIMG + cur.j * NKS * BLK;
    const float scale = p.sx[0] * p.sw[0];
    unsigned* tcnt = (unsigned*)(p.ws[NR - 1] + WS_TCNT);
    __amdgpu_buffer_rsrc_t ro = __builtin_amdgcn_make_buffer_rsrc(p.out[NR - 1], 0, 0, 0x00020000);
    int osh = -1;
    float osc = scale;
    const int vo = (lane >> 4) * (4 * N * 4) + (lane & 15) * 16;

    {
        for (int t = ((int)blockIdx.x - NCOMM) * 8 + wid; t < NTASK_IN; t += (NWG - NCOMM) * 8) {
            unsigned raw[16], off[4];
            Task k; u32x4 o[4];
            task_decode(rank, t, lane, k);
            task_load(p, k, lane, raw);
            task_tile(k, raw, o, off);
            if (k.only < 0 || owns_column(rank, k.only)) {
#pragma unroll
                for (int q = 0; q < 4; ++q) __builtin_amdgcn_raw_buffer_store_b128(o[q], ri, off[q], 0, AUX_SC1);
            }
        }
        WAIT_V(0);
        BAR;
        if (tid == 0) {
            const unsigned n = __hip_atomic_fetch_add((unsigned*)(p.ws[NR - 1] + WS_LCNT), 1u, __ATOMIC_RELAXED, __HIP_MEMORY_SCOPE_AGENT) + 1u;
            if (n == call * (unsigned)(NWG - NCOMM))
                __hip_atomic_store((unsigned*)(p.ws[NR - 1] + WS_PROG) + rank, pbase + NBLK, __ATOMIC_RELAXED, __HIP_MEMORY_SCOPE_SYSTEM);
            ctrl[0] = 0u;
            __builtin_amdgcn_fence(__ATOMIC_ACQUIRE, "");
            asm volatile("s_waitcnt vmcnt(0) lgkmcnt(0)" ::: "memory");
        }
    }
    GATE(cur.need0, cur.step, 0); GATE(cur.need0, cur.step, 1);
    BAR;
    STAGE(SB(0, 0), cB + KOFF(0)); STAGE(SA(0, 0), cA + KOFF(0)); STAGE(SB(0, 1), cB + KOFF(0) + HTB); STAGE(SA(0, 1), cA + KOFF(0) + HTB);
    if (wr == 1) BAR;
    WAIT_V(4); BAR;
    STAGE(SB(1, 0), cB + KOFF(1)); STAGE(SA(1, 0), cA + KOFF(1)); STAGE(SB(1, 1), cB + KOFF(1) + HTB);
    WAIT_V(6); BAR;
#pragma unroll 1
    for (int un = 0; un < 4; ++un) {
        tile_of(rank, xcd, rk + (un < 3 ? un + 1 : un) * nq, nxt);
        const int nA = nxt.i * NKS * BLK, nB = (int)AIMG + nxt.j * NKS * BLK;
        int t = 0;
        ITER(1, 0);
#pragma unroll 1
        for (t = 2; t < NKS - 2; t += 2) ITER(0, 0);
        ITER(0, 1);
        cur = nxt; cA = nA; cB = nB;
    }
    STORES(1, 0);
    STORES(1, 1);
    WAIT_V(0);
    if (wr == 0) BAR;
#undef ST8
#undef STORES
#undef ITER
#undef SA
#undef SB
#undef STAGE
#undef FRAG
#undef LDA
#undef LDB
#undef MMA
#undef WAIT_V
#undef WAIT_L
#undef BAR
#undef SCHED
#undef KOFF
#undef GATE
#undef POLL_ISSUE
#undef POLL_LANDED
}

__global__ void __launch_bounds__(512, 2) fused_gemm_allgather(const Params p) {
    extern __shared__ __attribute__((aligned(16))) unsigned char shm[];
    LAS unsigned char* lds = (LAS unsigned char*)shm;
    const int b = blockIdx.x;
    if (b < NCOMM) {
        const int wid = __builtin_amdgcn_readfirstlane((int)(threadIdx.x >> 6)), lane = threadIdx.x & 63;
        if (wid < TW) comm_tasks(p, lds + wid * 4096, b * TW + wid, lane);
        else if (b == 0 && wid == TW) comm_publish(p, lane);
    } else {
        const int xcd = b & 7;
        gemm_role(p, lds, xcd, (b >> 3) - (xcd < 6 ? 2 : 1));
    }
    __builtin_amdgcn_s_barrier();
    if (threadIdx.x == 0) {
        const unsigned ticket = __hip_atomic_fetch_add((unsigned*)(p.ws[NR - 1] + WS_TICKET), 1u, __ATOMIC_RELAXED, __HIP_MEMORY_SCOPE_AGENT) + 1u;
        if (ticket % NWG == 0u) __hip_atomic_store((unsigned*)(p.ws[NR - 1] + WS_EPOCH), ticket / NWG, __ATOMIC_RELAXED, __HIP_MEMORY_SCOPE_AGENT);
    }
}

}

extern "C" void kernel_launch(int rank, int nranks, ncclComm_t comm,
                              hipStream_t stream,
                              void* const* d_in, const int* in_sizes, int n_in,
                              void* d_out, int out_size,
                              const int* in_dtypes, int out_dtype,
                              void* const* d_out_all,
                              void* const* d_ws_all, size_t ws_bytes) {
    (void)comm; (void)in_sizes; (void)in_dtypes; (void)out_dtype; (void)d_out;
    if (nranks != NR || n_in != 4 || (unsigned)rank >= (unsigned)NR || ws_bytes < WS_NEED || out_size != M * N) {
        fprintf(stderr, "kernel_launch: unexpected problem shape\n");
        return;
    }
    Params p;
    p.x = (const unsigned char*)d_in[0];
    p.w = (const unsigned char*)d_in[1];
    p.sx = (const float*)d_in[2];
    p.sw = (const float*)d_in[3];
    for (int i = 0; i < NR; ++i) {
        p.ws[i] = (unsigned char*)d_ws_all[(rank + 1 + i) % NR];
        p.out[i] = (float*)d_out_all[(rank + 1 + i) % NR];
    }
    p.rank = rank;
    HIP_CHECK(hipFuncSetAttribute((const void*)fused_gemm_allgather, hipFuncAttributeMaxDynamicSharedMemorySize, LDS_BYTES));
    hipLaunchKernelGGL(fused_gemm_allgather, dim3(NWG), dim3(512), LDS_BYTES, stream, p);
    HIP_CHECK(hipGetLastError());
}
```
